```python
import numpy as np

kernel_name = "gpu_gemm_rs_m8192_k8192_n8192_bf16_silu"


ABSMAX_RTOL = 0.008

COMPUTE_DTYPE = "bfloat16"
EPILOGUE = "silu"
IN_QUANT = None
OUT_QUANT = None
IN_DTYPES = {"x": "bfloat16", "w_mat": "bfloat16"}
WORLD_SIZE = 8

def _to_bf16(x):
    u = np.ascontiguousarray(x, dtype=np.float32).view(np.uint32)
    return ((u + 0x7FFF + ((u >> 16) & 1)) & 0xFFFF0000).view(np.float32)

def setup_inputs():

    x = _to_bf16(np.random.randn(8192, 8192).astype(np.float32))
    w_mat = _to_bf16((np.random.randn(8192, 8192) / np.sqrt(8192)).astype(np.float32))
    return {"x": x, "w_mat": w_mat}

def _epilogue(y):
    return (y / (1.0 + np.exp(-np.clip(y, -60.0, 60.0)))).astype(np.float32)

def reference(x, w_mat):
    y = (x @ w_mat).astype(np.float32)
    return _epilogue(y)
```

```cpp
#include <hip/hip_runtime.h>
#include <rccl/rccl.h>
#include <cstdio>
#include <cstring>

#define HIP_CHECK(expr) do { hipError_t _e = (expr); if (_e != hipSuccess) { \
    fprintf(stderr, "HIP error %s at %s:%d\n", hipGetErrorString(_e), __FILE__, __LINE__); \
    return; } } while (0)

namespace {

#define LAS __attribute__((address_space(3)))
#define GAS __attribute__((address_space(1)))
#define CAS __attribute__((address_space(4)))
typedef GAS char gchar;
typedef GAS unsigned gu32;
typedef short bf16x8 __attribute__((ext_vector_type(8)));
typedef short s16x4 __attribute__((ext_vector_type(4)));
typedef float f32x4 __attribute__((ext_vector_type(4)));
typedef float f32x2 __attribute__((ext_vector_type(2)));
typedef __bf16 bf16x2 __attribute__((ext_vector_type(2)));
typedef unsigned u32x4 __attribute__((ext_vector_type(4)));

constexpr int NR = 8;
constexpr int GRID = 256;
#define NCOPY 32
constexpr int NC = NCOPY;
constexpr unsigned XP = 2048;
constexpr unsigned WP = 16384;
constexpr unsigned MiB = 1u << 20;
constexpr unsigned X_BYTES = 16 * MiB, W_BYTES = 16 * MiB, OUT_BYTES = 16 * MiB;
constexpr unsigned CTR_OFF = 0;
constexpr unsigned PROG_OFF = 32768;
constexpr unsigned TFLAG_OFF = 36864;
constexpr unsigned TMO_OFF = 40960;
constexpr unsigned CNT_OFF = 45056;
constexpr unsigned NAN_OFF = 49152;
constexpr unsigned DBG_OFF = 65536;
constexpr unsigned PAR_BYTES = 80 * MiB;
constexpr unsigned XST_OFF = 1 * MiB, WST_REL = 24 * MiB, YIN_REL = 48 * MiB;
constexpr unsigned YOUT_OFF = 161 * MiB;
constexpr unsigned WS_NEED = 193 * MiB;
constexpr unsigned TILE_B = 131072;
constexpr int NXW = 120;
constexpr int NYC = 64;
#ifndef CU_
#define CU_ 4
#endif
#define CW_ 2
#define NAP_UNIT 16
constexpr int BPC = 64 / CU_;
constexpr unsigned CALL_STRIDE = 8192;
constexpr unsigned long long DEADLINE_TICKS = 100000000ull;

constexpr int HTB = 16384;
constexpr int LDS_CTL = 8 * HTB;
constexpr int LDS_BYTES = LDS_CTL + 1024;

struct Params {
    unsigned long long x, w, out;
    unsigned long long ws[NR];
    int rank; int pad;
    unsigned char order[NR][NXW];
    unsigned char pos[NR][32][4];
    unsigned char yorder[NR][32];
    unsigned char ypos[NR][32];
    unsigned short items[GRID][2];
};

typedef const CAS Params* KP;
__device__ __forceinline__ int lds_byte(int r, int c) { const int st = (r >> 4) * 2 + (c >> 5), rr = r & 15, cc = c & 31, ob = rr * 64 + cc * 2; return st * 1024 + (ob ^ (((ob >> 9) & 1) << 5)); }
__device__ __forceinline__ void stage_rc(int b, int& R, int& C) { const int st = b / 1024, sb = b % 1024, swz = sb ^ (((sb >> 9) & 1) << 5); R = (st >> 1) * 16 + swz / 64; C = (st & 1) * 32 + (swz % 64) / 2; }
__device__ __forceinline__ int bswz(int k) { return ((k & 3) | (((k >> 3) & 1) << 2)) << 1; }
__device__ __forceinline__ unsigned pack_bf16(float lo, float hi) { const bf16x2 v = __builtin_convertvector((f32x2){lo, hi}, bf16x2); return __builtin_bit_cast(unsigned, v); }
__device__ __forceinline__ float bf_lo(unsigned u) { return __uint_as_float(u << 16); }
__device__ __forceinline__ float bf_hi(unsigned u) { return __uint_as_float(u & 0xffff0000u); }
__device__ __forceinline__ float silu(float y) {
    const float c = __builtin_amdgcn_fmed3f(y, -60.0f, 60.0f);
    return y * __builtin_amdgcn_rcpf(1.0f + __builtin_amdgcn_exp2f(c * -1.44269504089f));
}
__device__ __forceinline__ unsigned need_of(int chunks_done, unsigned call) {
    return call * CALL_STRIDE + (unsigned)(chunks_done * BPC);
}
#define RSRC(p, n) __builtin_amdgcn_make_buffer_rsrc((void*)(p), 0, (n), 0x00020000)
#define WAIT_V(n) asm volatile("s_waitcnt vmcnt(" #n ")" ::: "memory")
#define WAIT_L(n) asm volatile("s_waitcnt lgkmcnt(" #n ")" ::: "memory")
#define DRAIN_V do { asm volatile("" ::: "memory"); __builtin_amdgcn_s_waitcnt(0x0F70); asm volatile("s_waitcnt vmcnt(0)" ::: "memory"); } while (0)
#define BAR __builtin_amdgcn_s_barrier()
#define SCHED __builtin_amdgcn_sched_barrier(0)
#define CFENCE asm volatile("" ::: "memory")

#define STAMP(slot) do {} while (0)

__device__ __forceinline__ bool spin_expired(gu32* tmo, unsigned call, unsigned long long t0) {
    if ((unsigned)__builtin_amdgcn_readfirstlane(__hip_atomic_load(tmo, __ATOMIC_RELAXED, __HIP_MEMORY_SCOPE_AGENT)) == call ||
        __builtin_amdgcn_s_memrealtime() - t0 > DEADLINE_TICKS) {
        __hip_atomic_store(tmo, call, __ATOMIC_RELAXED, __HIP_MEMORY_SCOPE_AGENT);
        return true;
    }
    return false;
}
__device__ __forceinline__ void nap(unsigned gap) {
    unsigned n = min((gap * CU_) >> 5, 48u);
    do { __builtin_amdgcn_s_sleep(NAP_UNIT); } while (n--);
}
__device__ __forceinline__ unsigned link_progress(const gchar* ws0, int lane) {
    const u32x4 v = __builtin_amdgcn_raw_buffer_load_b128(RSRC(ws0, WS_NEED), PROG_OFF + lane * 16, 0, 17);
    unsigned m = min(min(v.x, v.y), min(v.z, v.w));
    if ((lane & 7) * 4 >= NC) m = 0xffffffffu;
    m = min(m, (unsigned)__shfl_xor((int)m, 1)); m = min(m, (unsigned)__shfl_xor((int)m, 2)); m = min(m, (unsigned)__shfl_xor((int)m, 4));
    return m;
}

struct CopySlot { unsigned voff, stride; };
template <int N> __device__ __forceinline__ void wait_vm() { asm volatile("s_waitcnt vmcnt(%0)" :: "i"(N) : "memory"); }
template <int u> __device__ __forceinline__ void pace() {
    if constexpr (u >= CW_) wait_vm<CW_ - 1 + ((u - CW_ <= CW_ - 2) ? 1 : 0)>();
    else wait_vm<CW_ + CU_ + ((CU_ - CW_ + u <= CW_ - 2) ? 1 : 0)>();
}
struct CopyCtx { unsigned long long x, w, ws; unsigned d, q, par, nxwb; int ord, myt, lane; };
__device__ __forceinline__ void copy_load(const CopyCtx c, bool dead, int g, u32x4 (&t)[CU_], CopySlot& s) {
    const unsigned qc = c.q & 1, qa = (c.q >> 1) & 1, qb = (c.q >> 2) & 1;
    const unsigned sbuf = ((c.d - 2) >> 1) * 8 * MiB, side = c.d & 1;
    const bool isy = g >= (int)c.nxwb;
    const int chunk = g / BPC, sub = g % BPC, yb = g - (int)c.nxwb;
    const unsigned e = (unsigned)__builtin_amdgcn_readlane(c.ord, chunk & 63) >> (((chunk >> 6) & 1) * 8), panel = e & 15, round = (e >> 4) & 3;
    const bool isw = e & 64;
    const unsigned rw = (round * 128 + sub * 2 * CU_) * WP + panel * 512, rx = (sub * 4 * CU_) * XP + round * 256;
    const unsigned tile = __builtin_amdgcn_readlane(c.myt, (yb / (2 * BPC)) & 31);
    const unsigned ry = (c.d * 32 + tile) * TILE_B + (yb % (2 * BPC)) * (CU_ * 1024);
    const unsigned sx = (((c.q ^ (panel >> 1)) * 1024 + qa * 512 + (panel & 1) * 256) * XP + qc * 1024) + rx;
    const unsigned so = isy ? (dead ? NAN_OFF : YOUT_OFF + ry) : isw ? qc * 512 * WP + qb * 8192 + rw : sx;
    const unsigned dn = c.par + (isy ? YIN_REL + ry : isw ? WST_REL + sbuf + side * 8192 + rw : sbuf + panel * 256 * XP + side * 1024 + rx);
    const unsigned sh = isy ? 6 : isw ? 5 : 4, pitch = isw ? WP : XP;
    const unsigned voff = (unsigned)(c.lane >> sh) * pitch + (c.lane & ((1 << sh) - 1)) * 16;
    const unsigned stride = isy ? 1024 : isw ? 2 * WP : 4 * XP;
    unsigned long long src = c.x; if (isw) src = c.w; if (isy) src = c.ws;
    const auto rs = RSRC(src, isy ? WS_NEED : X_BYTES);
#pragma unroll
    for (int u = 0; u < CU_; ++u) t[u] = __builtin_amdgcn_raw_buffer_load_b128(rs, voff, so + u * stride, 16);
    s.voff = voff + dn; s.stride = stride;
}
__device__ __forceinline__ unsigned copy_grab(gu32* cnt, int lane) { unsigned r = 0; if (lane == 0) r = __hip_atomic_fetch_add(cnt, 1u, __ATOMIC_RELAXED, __HIP_MEMORY_SCOPE_AGENT); return r; }
__device__ __forceinline__ void copy_publish(gu32* prog, unsigned v, int lane) { if (lane == 0) __hip_atomic_store(prog, v, __ATOMIC_RELAXED, __HIP_MEMORY_SCOPE_SYSTEM); }

__device__ __forceinline__ void copy_role(KP p, gchar* ws0, unsigned call) {
    const int wid = __builtin_amdgcn_readfirstlane(threadIdx.x >> 6), lane = threadIdx.x & 63;
    const int d = wid + 1;
    if ((unsigned)d >= (unsigned)NR) {
        if (blockIdx.x == 0 && lane < NR) __hip_atomic_store((gu32*)(ws0 + CNT_OFF + (((call + 1) & 1) * NR + lane) * 128), 0u, __ATOMIC_RELAXED, __HIP_MEMORY_SCOPE_AGENT);
        return;
    }
    gchar* wsd = (gchar*)p->ws[d];
    const auto rq = RSRC(wsd, WS_NEED);
    gu32* tmo = (gu32*)(ws0 + TMO_OFF);
    gu32* prog = (gu32*)(wsd + PROG_OFF + d * 128 + blockIdx.x * 4);
    int vzero = 0; asm volatile("" : "+v"(vzero));
    gu32* cnt = (gu32*)(ws0 + CNT_OFF + ((call & 1) * NR + d) * 128) + vzero;
    const int nxwb = d == 1 ? 0 : NXW * BPC, total = nxwb + NYC * BPC;
    const unsigned base = call * CALL_STRIDE;
    const int myt = p->yorder[d][lane & 31];
    CopyCtx c;
    c.x = p->x; c.w = p->w; c.ws = (unsigned long long)ws0; c.d = d; c.q = p->rank ^ d; c.par = XST_OFF + (call & 1) * PAR_BYTES; c.nxwb = nxwb;
    c.ord = p->order[d][lane] | (p->order[d][64 + lane < NXW ? 64 + lane : 0] << 8);
    c.myt = myt; c.lane = lane;
    bool dead = false;
#define grab() copy_grab(cnt, lane)
#define publish(v) copy_publish(prog, base + (v), lane)
#define load(g, t, s) copy_load(c, dead, g, t, s)
#define SEND1(tc, sc, u) if constexpr (u < CU_) { pace<u>(); if constexpr (u == CW_ - 1) { publish(cur); CFENCE; } \
        __builtin_amdgcn_raw_buffer_store_b128(tc[u], rq, sc.voff + u * sc.stride, 0, 17); CFENCE; }
#define CSTEP(tc, sc, tn, sn) { \
        const int nxt = __builtin_amdgcn_readfirstlane(drawn); \
        if (nxt >= limit) { \
            _Pragma("unroll") for (int u = 0; u < CU_; ++u) __builtin_amdgcn_raw_buffer_store_b128(tc[u], rq, sc.voff + u * sc.stride, 0, 17); \
            cur = nxt; break; } \
        CFENCE; drawn = grab(); CFENCE; load(nxt, tn, sn); CFENCE; \
        SEND1(tc, sc, 0) SEND1(tc, sc, 1) SEND1(tc, sc, 2) SEND1(tc, sc, 3) SEND1(tc, sc, 4) SEND1(tc, sc, 5) SEND1(tc, sc, 6) SEND1(tc, sc, 7) \
        cur = nxt; }
    int limit = nxwb;
    int cur = __builtin_amdgcn_readfirstlane(grab());
    while (cur < total) {
        if (cur >= limit) {
            publish(cur);
            const unsigned long long t0 = __builtin_amdgcn_s_memrealtime();
            for (;;) {
                const unsigned f = __hip_atomic_load((gu32*)(ws0 + TFLAG_OFF + d * 128) + myt, __ATOMIC_RELAXED, __HIP_MEMORY_SCOPE_AGENT);
                const unsigned long long nok = __ballot(f < call) & 0xffffffffull;
                limit = nxwb + (nok ? __builtin_ctzll(nok) : 32) * 2 * BPC;
                if (cur < limit) break;
                if (spin_expired(tmo, call, t0)) {
                    const auto rn = RSRC(ws0 + NAN_OFF, CU_ * 1024);
#pragma unroll
                    for (int u = 0; u < CU_; ++u) __builtin_amdgcn_raw_buffer_store_b128((u32x4){0x7fc07fc0u, 0x7fc07fc0u, 0x7fc07fc0u, 0x7fc07fc0u}, rn, lane * 16 + u * 1024, 0, 16);
                    DRAIN_V; dead = true; limit = total; break;
                }
                __builtin_amdgcn_s_sleep(NAP_UNIT);
            }
            CFENCE;
        }
        u32x4 ta[CU_], tb[CU_]; CopySlot sa, sb;
        unsigned drawn = grab(); CFENCE; load(cur, ta, sa);
        DRAIN_V;
        for (;;) { CSTEP(ta, sa, tb, sb) CSTEP(tb, sb, ta, sa) }
        DRAIN_V;
    }
#undef CSTEP
#undef SEND1
    publish(CALL_STRIDE - 1);
#undef grab
#undef publish
#undef load
    STAMP(9 + wid);
}

__device__ __forceinline__ int wait_steps(gchar* ws0, LAS unsigned char* lds, int want, unsigned call, int wid, int lane) {
    LAS unsigned* ctl = (LAS unsigned*)(lds + LDS_CTL);
    if (wid == 0) {
        const unsigned need = ctl[64 + lane], lnk = ctl[128 + lane];
        gu32* tmo = (gu32*)(ws0 + TMO_OFF);
        const unsigned long long t0 = __builtin_amdgcn_s_memrealtime();
        int upto;
        for (;;) {
            const unsigned m = link_progress(ws0, lane);
            const unsigned mine = (unsigned)__shfl((int)m, lnk * 8);
            const unsigned long long nok = __ballot(mine < need);
            upto = nok ? __builtin_ctzll(nok) : 64;
            if (upto > want) break;
            if (spin_expired(tmo, call, t0)) { upto = 64; break; }
            nap((unsigned)__builtin_amdgcn_readlane((int)(need - mine), upto));
        }
        __builtin_amdgcn_fence(__ATOMIC_ACQUIRE, "");
        if (lane == 0) ctl[0] = (unsigned)upto;
        asm volatile("s_waitcnt vmcnt(0) lgkmcnt(0)" ::: "memory");
    }
    CFENCE; BAR; CFENCE;
    const int r = __builtin_amdgcn_readfirstlane(ctl[0]);
    CFENCE;
    return r;
}

__device__ __forceinline__ void gemm_role(KP p, gchar* ws0, LAS unsigned char* lds, unsigned call) {
    const gchar* px = (const gchar*)p->x; const gchar* pw = (const gchar*)p->w;
    const int tid = threadIdx.x, wid = __builtin_amdgcn_readfirstlane(tid >> 6), lane = tid & 63, wr = wid >> 2, wc = wid & 3, fr = lane & 15, fq = lane >> 4;
    const unsigned rank = p->rank, ra = (rank >> 1) & 1, rb = (rank >> 2) & 1;
    const gchar* xst = ws0 + XST_OFF + (call & 1) * PAR_BYTES;
    const gchar* wst = xst + WST_REL;
    LAS unsigned* ctl = (LAS unsigned*)(lds + LDS_CTL);
    unsigned voffA, voffB;
    { int R, C; stage_rc(tid * 16, R, C); voffA = (unsigned)R * XP + C * 2;
      const int k = wid * 4 + (lane >> 4); voffB = (unsigned)k * WP + (((lane & 15) ^ bswz(k)) * 16); }
    const unsigned ldsw = (unsigned)wid * 1024u;
    const int aoff = lds_byte(wr * 64 + fr, fq * 8);
    unsigned boff[2];
    { const int k = 8 * fq + ((lane >> 2) & 3), pp = lane & 3;
#pragma unroll
      for (int n = 0; n < 2; ++n) boff[n] = (unsigned)(size_t)lds + 4 * HTB + k * 256 + (((wc * 4 + n * 2 + (pp >> 1)) ^ bswz(k)) * 16) + (pp & 1) * 8; }
    constexpr size_t KA = 128, HA = 128 * XP, PA = 64 * XP;
    constexpr size_t KB = 64 * WP, HB = 256, PB = 32 * WP;
#define SA(b, h) (((b) * 2 + (h)) * HTB)
#define SB(b, h) ((4 + (b) * 2 + (h)) * HTB)
#define STAGE(bufoff, gbase, voff, pstep) do { _Pragma("unroll") for (int _i = 0; _i < 2; ++_i) \
        __builtin_amdgcn_global_load_lds((const gu32*)((const gchar*)(gbase) + _i * (pstep) + (voff)), (LAS unsigned*)(lds + (bufoff) + ldsw + _i * 8192), 16, 0, 0); } while (0)
#define STA(b, h, g) STAGE(SA(b, h), (g) + (h) * HA, voffA, PA)
#define STB(b, h, g) STAGE(SB(b, h), (g) + (h) * HB, voffB, PB)
#define LDA(dst, b, h) do { _Pragma("unroll") for (int m = 0; m < 4; ++m) _Pragma("unroll") for (int k = 0; k < 2; ++k) dst[m][k] = *(const LAS bf16x8*)(lds + SA(b, h) + aoff + m * 2048 + k * 1024); } while (0)
#define TRRD(dst, addr, off) asm volatile("ds_read_b64_tr_b16 %0, %1 offset:%2" : "=&v"(dst) : "v"(addr), "i"(off) : "memory")
#define LDB(dst, b, h) do { _Pragma("unroll") for (int n = 0; n < 2; ++n) _Pragma("unroll") for (int k = 0; k < 2; ++k) { s16x4 _lo, _hi; \
        TRRD(_lo, boff[n], ((b) * 2 + (h)) * HTB + k * 8192); TRRD(_hi, boff[n], ((b) * 2 + (h)) * HTB + k * 8192 + 1024); \
        dst[n][k] = __builtin_shufflevector(_lo, _hi, 0, 1, 2, 3, 4, 5, 6, 7); } } while (0)
#define MMA(ai, bj, At, Bt) do { __builtin_amdgcn_s_setprio(1); _Pragma("unroll") for (int m = 0; m < 4; ++m) _Pragma("unroll") for (int n = 0; n < 2; ++n) _Pragma("unroll") for (int k = 0; k < 2; ++k) \
        acc[ai][bj][m][n] = __builtin_amdgcn_mfma_f32_16x16x32_bf16(Bt[n][k], At[m][k], acc[ai][bj][m][n], 0, 0, 0); __builtin_amdgcn_s_setprio(0); } while (0)

    for (int it = 0; it < 2; ++it) {
        const unsigned item = p->items[blockIdx.x][it];
        if (!(item & 0x8000u)) break;
        const bool fin = item & 0x4000u;
        const unsigned rp = (item >> 8) & 15, cp = item & 31;
        const unsigned dq = fin ? 1u : rp >> 1;
        const int nown = (fin || dq != 0) ? 8 : 0;
        const int nd = dq >= 2 ? 5 : 6;
        const int nsteps = fin ? 8 : nown + 4 * nd;
        const unsigned row0 = fin ? rank * 1024 + rp * 256 : (rank ^ dq) * 1024 + ra * 512 + (rp & 1) * 256;
        const unsigned colb = fin ? cp * 512 : rb * 8192 + cp * 512;
        const gchar* ownA = px + (size_t)row0 * XP; const gchar* ownB = pw + colb;
        const gchar* remA = xst + (size_t)rp * 256 * XP; const gchar* remB = wst + cp * 512;
        STAMP(1 + it * 4);
        __syncthreads();
        if (wid == 0) {
            unsigned need = 0, lnk = 0;
            const int idx = lane - nown;
            if (idx >= 0 && lane < nsteps) {
                const int round = idx / nd; int dd = 2 + idx - round * nd; if (dq >= 2 && dd >= (int)dq) ++dd;
                const int c = max((int)p->pos[dd][rp][round], (int)p->pos[dd][16 + cp][round]) + 1;
                need = need_of(c, call); lnk = dd;
            }
            ctl[64 + lane] = need; ctl[128 + lane] = lnk;
            WAIT_L(0);
        }
        int sd = 1, sround = 0;
        auto next_remote = [&](const gchar*& a, const gchar*& b) {
            ++sd; if (sd == (int)dq) ++sd;
            if (sd > 7) { ++sround; sd = dq == 2 ? 3 : 2; }
            const unsigned bo = (unsigned)((sd - 2) >> 1) * 8 * MiB, side = sd & 1;
            a = remA + (bo + side * 1024 + sround * 256);
            b = remB + (bo + side * 8192 + (unsigned)sround * 128 * WP);
        };
        int upto = nown;
        const gchar* cA; const gchar* cB;
        if (nown) { cA = ownA; cB = ownB; }
        else { upto = wait_steps(ws0, lds, 0, call, wid, lane); next_remote(cA, cB); }

        f32x4 acc[2][2][4][2];
#pragma unroll
        for (int a = 0; a < 2; ++a)
#pragma unroll
            for (int b = 0; b < 2; ++b)
#pragma unroll
                for (int m = 0; m < 4; ++m)
#pragma unroll
                    for (int n = 0; n < 2; ++n) acc[a][b][m][n] = (f32x4){0.f, 0.f, 0.f, 0.f};
        bf16x8 At[4][2], B0[2][2], B1[2][2];
        STB(0, 0, cB); STA(0, 0, cA); STB(0, 1, cB); STA(0, 1, cA);
        if (wr == 1) BAR;
        WAIT_V(4); BAR;
        STB(1, 0, cB + KB); STA(1, 0, cA + KA); STB(1, 1, cB + KB);
        WAIT_V(6); BAR;
        for (int s = 0; s < nsteps; ++s) {
            const gchar* nA = cA; const gchar* nB = cB;
            if (s + 1 < nsteps) {
                if (s + 1 < nown) { nA = cA + 2 * KA; nB = cB + 2 * KB; }
                else {
                    if (s + 1 >= upto) upto = wait_steps(ws0, lds, s + 1, call, wid, lane);
                    next_remote(nA, nB);
                }
            }
            LDB(B0, 0, 0); SCHED; LDA(At, 0, 0); STA(1, 1, cA + KA);
            WAIT_L(8); BAR; WAIT_L(0); SCHED; MMA(0, 0, At, B0); BAR; SCHED;
            LDB(B1, 0, 1); STB(0, 0, nB);
            BAR; WAIT_L(0); SCHED; MMA(0, 1, At, B1); BAR;
            LDA(At, 0, 1); STA(0, 0, nA);
            BAR; WAIT_L(0); SCHED; MMA(1, 0, At, B0); BAR; SCHED;
            STB(0, 1, nB);
            WAIT_V(6); BAR; MMA(1, 1, At, B1); BAR;
            LDB(B0, 1, 0); SCHED; LDA(At, 1, 0); STA(0, 1, nA);
            WAIT_L(8); BAR; WAIT_L(0); SCHED; MMA(0, 0, At, B0); BAR; SCHED;
            LDB(B1, 1, 1); STB(1, 0, nB + KB);
            BAR; WAIT_L(0); SCHED; MMA(0, 1, At, B1); BAR;
            LDA(At, 1, 1); STA(1, 0, nA + KA);
            BAR; WAIT_L(0); SCHED; MMA(1, 0, At, B0); BAR; SCHED;
            STB(1, 1, nB + KB);
            WAIT_V(6); BAR; MMA(1, 1, At, B1); BAR;
            cA = nA; cB = nB;
        }
        WAIT_V(0);
        if (wr == 0) BAR;
        BAR;
        STAMP(2 + it * 4);
        gu32* tmo = (gu32*)(ws0 + TMO_OFF);
        const auto rl = RSRC(ws0, WS_NEED);
        const unsigned yin = XST_OFF + (call & 1) * PAR_BYTES + YIN_REL;
        int tz = threadIdx.x; asm volatile("" : "+v"(tz));
        const unsigned fragoff = (unsigned)(tz >> 6) * 16384 + (tz & 63) * 16;
        if (!fin) {
            const unsigned tile = (rp & 1) * 16 + cp;
            const unsigned dst = (dq == 0 ? yin : YOUT_OFF) + (dq * 32 + tile) * TILE_B;
            const auto rt = RSRC(ws0 + dst, TILE_B);
            const bool bad = (unsigned)__builtin_amdgcn_readfirstlane(__hip_atomic_load(tmo, __ATOMIC_RELAXED, __HIP_MEMORY_SCOPE_AGENT)) == call;
#pragma unroll
            for (int ai = 0; ai < 2; ++ai)
#pragma unroll
                for (int m = 0; m < 4; ++m)
#pragma unroll
                    for (int bj = 0; bj < 2; ++bj) {
                        const f32x4 v0 = acc[ai][bj][m][0], v1 = acc[ai][bj][m][1];
                        u32x4 o = {pack_bf16(v0[0], v0[1]), pack_bf16(v0[2], v0[3]), pack_bf16(v1[0], v1[1]), pack_bf16(v1[2], v1[3])};
                        if (bad) o = (u32x4){0x7fc07fc0u, 0x7fc07fc0u, 0x7fc07fc0u, 0x7fc07fc0u};
                        __builtin_amdgcn_raw_buffer_store_b128(o, rt, fragoff + (ai * 8 + m * 2 + bj) * 1024, 0, 16);
                    }
            WAIT_V(0);
            __syncthreads();
            if (threadIdx.x == 0) __hip_atomic_store((gu32*)(ws0 + TFLAG_OFF + dq * 128) + tile, call, __ATOMIC_RELAXED, __HIP_MEMORY_SCOPE_AGENT);
        } else {
            const unsigned tile = (rp & 1) * 16 + (cp & 15);
            const unsigned d0 = (((cp >> 4) << 2) | ((rp >> 1) << 1)) ^ rank, d1 = d0 ^ 1;
            if (wid == 0) {
                const unsigned dm = lane & 1 ? d1 : d0;
                const unsigned need = need_of((dm == 1 ? 0 : NXW) + 2 * p->ypos[dm][tile] + 2, call);
                const unsigned long long t0 = __builtin_amdgcn_s_memrealtime();
                for (;;) {
                    const unsigned m = link_progress(ws0, lane);
                    const unsigned f = __hip_atomic_load((gu32*)(ws0 + TFLAG_OFF) + tile, __ATOMIC_RELAXED, __HIP_MEMORY_SCOPE_AGENT);
                    const unsigned mine = (unsigned)__shfl((int)m, dm * 8);
                    const bool nok = dm == 0 ? f < call : mine < need;
                    if (!__ballot(nok)) break;
                    if (spin_expired(tmo, call, t0)) break;
                    const int gap = (nok && dm != 0) ? (int)(need - mine) : 0;
                    nap((unsigned)max(__builtin_amdgcn_readlane(gap, 0), __builtin_amdgcn_readlane(gap, 1)));
                }
                __builtin_amdgcn_fence(__ATOMIC_ACQUIRE, "");
                WAIT_V(0);
            }
            __syncthreads();
            STAMP(3 + it * 4);
            const bool bad = (unsigned)__builtin_amdgcn_readfirstlane(__hip_atomic_load(tmo, __ATOMIC_RELAXED, __HIP_MEMORY_SCOPE_AGENT)) == call;
            const unsigned s0 = yin + (d0 * 32 + tile) * TILE_B, s1 = yin + (d1 * 32 + tile) * TILE_B;
            const int lz = tz & 63, wz = tz >> 6, fqz = lz >> 4;
            const unsigned ooff = (unsigned)(rp * 256 + (wz >> 2) * 64 + (lz & 15)) * WP + cp * 512 + (wz & 3) * 64 + (fqz & 1) * 32 + (fqz >> 1) * 16;
            const auto ro = RSRC((gchar*)p->out, OUT_BYTES);
#pragma unroll
            for (int ai = 0; ai < 2; ++ai) {
                u32x4 pa[8], pb[8];
                if (!bad) {
#pragma unroll
                    for (int i = 0; i < 8; ++i) { pa[i] = __builtin_amdgcn_raw_buffer_load_b128(rl, fragoff, s0 + (ai * 8 + i) * 1024, 0); pb[i] = __builtin_amdgcn_raw_buffer_load_b128(rl, fragoff, s1 + (ai * 8 + i) * 1024, 0); }
                } else {
#pragma unroll
                    for (int i = 0; i < 8; ++i) { pa[i] = (u32x4){0x7fc07fc0u, 0x7fc07fc0u, 0x7fc07fc0u, 0x7fc07fc0u}; pb[i] = pa[i]; }
                }
#pragma unroll
                for (int m = 0; m < 4; ++m)
#pragma unroll
                    for (int bj = 0; bj < 2; ++bj) {
                        const u32x4 a = pa[m * 2 + bj], b = pb[m * 2 + bj];
                        const f32x4 v0 = acc[ai][bj][m][0], v1 = acc[ai][bj][m][1];
                        const unsigned o0 = pack_bf16(silu(v0[0] + bf_lo(a.x) + bf_lo(b.x)), silu(v0[1] + bf_hi(a.x) + bf_hi(b.x)));
                        const unsigned o1 = pack_bf16(silu(v0[2] + bf_lo(a.y) + bf_lo(b.y)), silu(v0[3] + bf_hi(a.y) + bf_hi(b.y)));
                        const unsigned o2 = pack_bf16(silu(v1[0] + bf_lo(a.z) + bf_lo(b.z)), silu(v1[1] + bf_hi(a.z) + bf_hi(b.z)));
                        const unsigned o3 = pack_bf16(silu(v1[2] + bf_lo(a.w) + bf_lo(b.w)), silu(v1[3] + bf_hi(a.w) + bf_hi(b.w)));
                        const auto r0 = __builtin_amdgcn_permlane16_swap(o0, o2, false, false);
                        const auto r1 = __builtin_amdgcn_permlane16_swap(o1, o3, false, false);
                        const u32x4 o = {r0[0], r1[0], r0[1], r1[1]};
                        __builtin_amdgcn_raw_buffer_store_b128(o, ro, ooff + (unsigned)(ai * 128 + m * 16) * WP + bj * 256, 0, 0);
                    }
                CFENCE;
            }
        }
        STAMP(4 + it * 4);
    }
}

__global__ void __launch_bounds__(512, 2) fused_gemm_rs(const Params args) {
    extern __shared__ __attribute__((aligned(16))) unsigned char shm[];
    KP p = (KP)__builtin_amdgcn_kernarg_segment_ptr();
    gchar* ws0 = (gchar*)p->ws[0];
    gu32* ctr = (gu32*)(ws0 + CTR_OFF + blockIdx.x * 64);
    STAMP(0);
    const unsigned call = (unsigned)__builtin_amdgcn_readfirstlane(__hip_atomic_load(ctr, __ATOMIC_RELAXED, __HIP_MEMORY_SCOPE_AGENT)) + 1u;
    WAIT_V(0);
    __syncthreads();
    if (threadIdx.x == 0) __hip_atomic_store(ctr, call, __ATOMIC_RELAXED, __HIP_MEMORY_SCOPE_AGENT);
    if (blockIdx.x < (unsigned)NC) copy_role(p, ws0, call);
    else gemm_role(p, ws0, (LAS unsigned char*)shm, call);
}

void build_tables(Params& p) {
    constexpr int NP = 12, NQ = 20;
    static const int prio[NP] = {2, 16, 4, 17, 6, 18, 8, 19, 10, 12, 14, 3};
    static const int rest[NQ] = {20, 5, 21, 7, 22, 9, 23, 11, 24, 13, 25, 15, 26, 27, 28, 29, 30, 31, 0, 1};
    memset(p.order, 0, sizeof p.order); memset(p.pos, 0, sizeof p.pos);
    for (int d = 2; d < NR; ++d) {
        int n = 0;
        for (int ph = 0; ph < 2; ++ph)
            for (int r = 0; r < 4; ++r) {
                const int* l = ph ? rest : prio; const int cnt = ph ? NQ : NP;
                for (int i = 0; i < cnt; ++i) {
                    const int id = l[i];
                    if (id < 16 && (id >> 1) == d) continue;
                    p.order[d][n] = (unsigned char)(((id >> 4) << 6) | (r << 4) | (id & 15));
                    p.pos[d][id][r] = (unsigned char)n; ++n;
                }
            }
    }
    int late[32];
    for (int i = 0; i < NP; ++i) late[prio[i]] = -1;
    for (int i = 0; i < NQ; ++i) late[rest[i]] = i;
    for (int d = 0; d < NR; ++d) {
        int key[32];
        for (int t = 0; t < 32; ++t) { const int a = late[d * 2 + (t >> 4)], b = late[16 + (t & 15)]; key[t] = ((a > b ? a : b) + 1) * 32 + t; }
        for (int s = 0; s < 32; ++s) { int best = -1; for (int t = 0; t < 32; ++t) if (key[t] >= 0 && (best < 0 || key[t] < key[best])) best = t;
            p.yorder[d][s] = (unsigned char)best; p.ypos[d][best] = (unsigned char)s; key[best] = -1; }
    }
    memset(p.items, 0, sizeof p.items);
    const int per = (GRID - NC) / 8;
    int selfq[32], nself = 0; bool used[32] = {false};
    for (int x = 0; x < 8; ++x) {
        int slot = 0;
        for (int d = 1; d < NR; ++d) for (int c = 0; c < 4; ++c) { const int b = NC + (slot++) * 8 + x; p.items[b][0] = (unsigned short)(0x8000 | ((d * 2 + (x & 1)) << 8) | ((x >> 1) * 4 + c)); }
        for (int c = 3; slot < per; --c) { const int b = NC + (slot++) * 8 + x; const int col = (x >> 1) * 4 + c; p.items[b][0] = (unsigned short)(0x8000 | ((x & 1) << 8) | col); used[(x & 1) * 16 + col] = true; }
    }
    for (int t = 0; t < 32; ++t) if (!used[t]) selfq[nself++] = t;
    int fin = 0, sq = 0;
    int keyb[GRID];
    for (int b = 0; b < GRID; ++b) {
        keyb[b] = -1;
        if (b < NC) continue;
        const int rp = (p.items[b][0] >> 8) & 15, cp = p.items[b][0] & 31;
        const int a = late[rp], c = late[16 + cp], k = a > c ? a : c;
        if (k < 0 && sq < nself) { const int t = selfq[sq++]; p.items[b][1] = (unsigned short)(0x8000 | ((t >> 4) << 8) | (t & 15)); }
        else keyb[b] = (k + 1) * GRID + b;
    }
    for (; sq < nself; ++sq) { int best = -1; for (int b = 0; b < GRID; ++b) if (keyb[b] >= 0 && (best < 0 || keyb[b] < keyb[best])) best = b;
        const int t = selfq[sq]; p.items[best][1] = (unsigned short)(0x8000 | ((t >> 4) << 8) | (t & 15)); keyb[best] = -1; }
    bool chosen[GRID] = {false};
    for (int i = 0; i < 128; ++i) { int best = -1; for (int b = 0; b < GRID; ++b) if (keyb[b] >= 0 && (best < 0 || keyb[b] < keyb[best])) best = b; chosen[best] = true; keyb[best] = -1; }
    for (int x = 0; x < 8; ++x) for (int b = NC + x; b < GRID; b += 8) if (chosen[b]) { p.items[b][1] = (unsigned short)(0xC000 | ((fin & 3) << 8) | (fin >> 2)); ++fin; }
}

}

extern "C" void kernel_launch(int rank, int nranks, ncclComm_t comm, hipStream_t stream,
                              void* const* d_in, const int* in_sizes, int n_in,
                              void* d_out, int out_size, const int* in_dtypes, int out_dtype,
                              void* const* d_out_all, void* const* d_ws_all, size_t ws_bytes) {
    (void)comm; (void)in_dtypes; (void)out_dtype; (void)d_out_all;
    if (nranks != NR || (unsigned)rank >= (unsigned)NR || n_in != 2 || in_sizes[0] != 8192 * 1024 || in_sizes[1] != 1024 * 8192 ||
        out_size != 1024 * 8192 || ws_bytes < WS_NEED) { fprintf(stderr, "unexpected problem shape\n"); return; }
    Params p;
    p.x = (unsigned long long)d_in[0]; p.w = (unsigned long long)d_in[1]; p.out = (unsigned long long)d_out;
    for (int d = 0; d < NR; ++d) p.ws[d] = (unsigned long long)d_ws_all[rank ^ d];
    p.rank = rank; p.pad = 0;
    build_tables(p);
    HIP_CHECK(hipFuncSetAttribute((const void*)fused_gemm_rs, hipFuncAttributeMaxDynamicSharedMemorySize, LDS_BYTES));
    hipLaunchKernelGGL(fused_gemm_rs, dim3(GRID), dim3(512), LDS_BYTES, stream, p);
    HIP_CHECK(hipGetLastError());
}
```

```cpp
#include <hip/hip_runtime.h>
#include <rccl/rccl.h>
#include <cstdio>
#include <cstring>
#include <cstddef>

#define HIP_CHECK(expr) do { hipError_t _e = (expr); if (_e != hipSuccess) { \
    fprintf(stderr, "HIP error %s at %s:%d\n", hipGetErrorString(_e), __FILE__, __LINE__); \
    return; } } while (0)

namespace {

#define LAS __attribute__((address_space(3)))
#define GAS __attribute__((address_space(1)))
#define CAS __attribute__((address_space(4)))
typedef GAS char gchar;
typedef GAS unsigned gu32;
typedef short bf16x8 __attribute__((ext_vector_type(8)));
typedef short s16x4 __attribute__((ext_vector_type(4)));
typedef float f32x4 __attribute__((ext_vector_type(4)));
typedef float f32x2 __attribute__((ext_vector_type(2)));
typedef __bf16 bf16x2 __attribute__((ext_vector_type(2)));
typedef unsigned u32x4 __attribute__((ext_vector_type(4)));

constexpr int NR = 8;
constexpr int GRID = 256;
constexpr unsigned XP = 2048;
constexpr unsigned WP = 16384;
constexpr unsigned MiB = 1u << 20;
constexpr unsigned X_BYTES = 16 * MiB, OUT_BYTES = 16 * MiB;
constexpr unsigned CTR_OFF = 0;
constexpr unsigned LS = 384;
constexpr unsigned TFLAG_OFF = 16384;
constexpr unsigned TMO_OFF = 20480;
constexpr unsigned EPROG_OFF = 24576;
constexpr unsigned EQ_OFF = 25600;
constexpr unsigned SUM_OFF = 32768;
constexpr unsigned LIM_OFF = SUM_OFF + 8 * LS;
constexpr unsigned PLOC_OFF = SUM_OFF + 17 * LS;
constexpr unsigned RMSG_OFF = SUM_OFF + 25 * LS;
constexpr unsigned CNT_OFF = SUM_OFF + 33 * LS;
constexpr unsigned HEAD_OFF = SUM_OFF + 49 * LS;
constexpr unsigned YIN_OFF = 1 * MiB;
constexpr unsigned PAR_BYTES = 112 * MiB;
constexpr unsigned XST_OFF = 33 * MiB, STG_BYTES = 48 * MiB, YOUT_REL = 48 * MiB, XEX_REL = 80 * MiB;
constexpr unsigned WS_NEED = 257 * MiB;
constexpr unsigned OOB = 0x80000000u;
constexpr unsigned TILE_B = 131072;
constexpr int MAXE = 192;
constexpr int CU_ = 4;
constexpr int NPUSH = 3;
constexpr int MARK_LEAD = 8;
constexpr int PULL_LEAD = 64;
constexpr unsigned PULL_DELAY = 560;
constexpr unsigned LAZY_TICKS = 2000;
constexpr int OWN_MAIN = 5;
constexpr int BPC = 64 / CU_, BPU = BPC / 2;
constexpr unsigned CALL_STRIDE = 4096;
constexpr unsigned long long DEADLINE_TICKS = 100000000ull;

constexpr int HTB = 16384;
constexpr int LDS_CTL = 8 * HTB;
constexpr int LDS_BYTES = LDS_CTL + 2048;

constexpr int NC = 32;
constexpr int XR = 3;
static const unsigned short SCHED_NENT[NR] = {0, 148, 162, 162, 162, 162, 162, 162};
static const unsigned SCHED_ENT[NR][MAXE] = {
    {
        0x00000000},
    {
        0x00000000, 0x00000108, 0x00000210, 0x00000318, 0x40000000, 0x40000100, 0x40000200, 0x40000300, 0x40000400, 0x40000500, 0x40000600, 0x40000700,
        0x40000800, 0x40000900, 0x40000a00, 0x40000b00, 0x40000c00, 0x40000d00, 0x40000e00, 0x40000f00, 0x00001000, 0x00001108, 0x00001210, 0x00001318,
        0x40001000, 0x40001100, 0x40001200, 0x40001300, 0x40001400, 0x40001500, 0x40001600, 0x40001700, 0x40001800, 0x40001900, 0x40001a00, 0x40001b00,
        0x40001c00, 0x40001d00, 0x40001e00, 0x40001f00, 0xc00c0301, 0xc00c0b03, 0xc00c1305, 0xc00c1b07, 0xc0040101, 0xc0040903, 0xc0041105, 0xc0041907,
        0xc0050141, 0xc0050943, 0xc0051145, 0xc0051947, 0xc00300c1, 0xc00308c3, 0xc00310c5, 0xc00318c7, 0xc00320c9, 0xc00328cb, 0xc0042109, 0xc004290b,
        0xc0052149, 0xc005294b, 0xc004310d, 0xc004390f, 0xc005314d, 0xc005394f, 0xc00330cd, 0xc00338cf, 0xc00340d1, 0xc00348d3, 0xc0044111, 0xc0044913,
        0xc00b12c5, 0xc00b1ac7, 0xc00b22c9, 0xc00b2acb, 0xc00b52d5, 0xc00b5ad7, 0xc00350d5, 0xc00358d7, 0xc0045115, 0xc0045917, 0xc0055155, 0xc0055957,
        0xc00e5395, 0xc00e5b97, 0xc00e6399, 0xc00e6b9b, 0xc0054151, 0xc0054953, 0xc00360d9, 0xc00368db, 0xc0046119, 0xc004691b, 0xc0056159, 0xc005695b,
        0xc0085215, 0xc0085a17, 0xc00370dd, 0xc00378df, 0xc004711d, 0xc004791f, 0xc005715d, 0xc005795f, 0xc00380e1, 0xc00388e3, 0xc0048121, 0xc0048923,
        0xc0058161, 0xc0058963, 0xc00390e5, 0xc00398e7, 0xc0049125, 0xc0049927, 0xc0059165, 0xc0059967, 0xc003b0ed, 0xc003b8ef, 0xc004b12d, 0xc004b92f,
        0xc0094251, 0xc0094a53, 0xc0098261, 0xc0098a63, 0xc005b16d, 0xc005b8e8, 0xc003a4ea, 0xc004a129, 0xc004a968, 0xc005a56a, 0xc003c0f1, 0xc003c930,
        0xc004c532, 0xc005c171, 0xc003d0f5, 0xc004d135, 0xc005d175, 0xc003f0fd, 0xc004f13d, 0xc005f17d, 0xc003e0f9, 0xc004e139, 0xc005e179, 0xc005c8f6,
        0xc004d976, 0xc003f93e, 0xc005f8fa, 0xc004e97a},
    {
        0x00000cd0, 0x40000000, 0x40001000, 0x00001cd0, 0x40002000, 0x40003000, 0x00000428, 0x40000100, 0x00000530, 0x00001428, 0x00002428, 0x00000320,
        0x40001100, 0x00003428, 0x00002cd0, 0x00001530, 0x40002100, 0x00001320, 0x40000200, 0x800002c0, 0x800002e0, 0x00000ba8, 0x00002530, 0x00003cd0,
        0x40000500, 0x800005c0, 0x800005e0, 0x40003100, 0x40001200, 0x00002320, 0x40000300, 0x00001ba8, 0x00003530, 0x40001300, 0x40000400, 0x40002200,
        0x40001500, 0x40002300, 0x80002380, 0x800023a0, 0x00003320, 0x00000ef0, 0x40003300, 0x40001400, 0x40000600, 0x00001ef0, 0x40003200, 0x00002ba8,
        0x40002400, 0x800024c0, 0x800024e0, 0x40002500, 0x40000700, 0x40001600, 0x00002ef0, 0x40003400, 0x40001700, 0x00003ef0, 0x40002600, 0x80002680,
        0x800026a0, 0x00000aa0, 0x00000880, 0x40000800, 0x00003ba8, 0x00000ff8, 0x40000900, 0x40002700, 0x40003500, 0x40000b00, 0x00001aa0, 0x00001880,
        0x40001800, 0x00002aa0, 0x40001900, 0x40000a00, 0x40003600, 0x40003700, 0x00001ff8, 0x00002880, 0x40000c00, 0x40001b00, 0x40002800, 0x00000dd8,
        0x00003aa0, 0x00000988, 0x00000778, 0x40001a00, 0x00003880, 0x40002900, 0x40000d00, 0x00001dd8, 0x40000f00, 0x40003800, 0x40002b00, 0x00001988,
        0x40002a00, 0x00002ff8, 0x40001c00, 0x00002dd8, 0x00001778, 0x40003900, 0x40000e00, 0x00002988, 0x40003a00, 0x00003dd8, 0x40003b00, 0x00003ff8,
        0x40001d00, 0x40002c00, 0x80002c80, 0x80002ca0, 0x00003988, 0x00002778, 0x40001f00, 0x40001e00, 0x00003778, 0x40003c00, 0x40002d00, 0x40002e00,
        0x40002f00, 0x80002f80, 0x80002fa0, 0x40003d00, 0x40003e00, 0x00000000, 0x40003f00, 0x00001000, 0x00002000, 0x00000210, 0x00001210, 0x00003000,
        0x00000108, 0x00001108, 0x00002108, 0x00002210, 0x00003108, 0x00003210, 0xc0060181, 0xc0060984, 0xc0061586, 0xc0062189, 0xc006298c, 0xc006358e,
        0xc0064191, 0xc0064994, 0xc0065596, 0xc0066199, 0xc006719d, 0xc00681a1, 0xc00691a5, 0xc006a1a9, 0xc006b1ad, 0xc006c1b1, 0xc006d1b5, 0xc006f1bd,
        0xc006e1b9, 0xc006699e, 0xc00689a6, 0xc006a9ae, 0xc006c9b6, 0xc006f9ba},
    {
        0x00000cd0, 0x40000000, 0x40001000, 0x00001cd0, 0x40002000, 0x40003000, 0x00000428, 0x40000100, 0x00000530, 0x00001428, 0x00002428, 0x00000320,
        0x40001100, 0x00003428, 0x00002cd0, 0x00001530, 0x40002100, 0x00001320, 0x40000200, 0x00000ba8, 0x00002530, 0x00003cd0, 0x40000500, 0x40003100,
        0x40001200, 0x00002320, 0x40000300, 0x00001ba8, 0x00003530, 0x40001300, 0x40000400, 0x40002200, 0x40001500, 0x800015e0, 0x800015c0, 0x40002300,
        0x00003320, 0x00000ef0, 0x40003300, 0x40001400, 0x40000600, 0x00001ef0, 0x40003200, 0x00002ba8, 0x40002400, 0x40002500, 0x40000700, 0x40001600,
        0x800016a0, 0x80001680, 0x00002ef0, 0x40003400, 0x40001700, 0x800017a0, 0x80001780, 0x00003ef0, 0x40002600, 0x00000aa0, 0x00000880, 0x40000800,
        0x00003ba8, 0x00000ff8, 0x40000900, 0x40002700, 0x40003500, 0x40000b00, 0x00001aa0, 0x00001880, 0x40001800, 0x00002aa0, 0x40001900, 0x40000a00,
        0x40003600, 0x00000658, 0x40003700, 0x00001ff8, 0x00002880, 0x40000c00, 0x40001b00, 0x40002800, 0x00000dd8, 0x00003aa0, 0x00000988, 0x40001a00,
        0x00003880, 0x40002900, 0x40000d00, 0x00001dd8, 0x40000f00, 0x40003800, 0x800038a0, 0x80003880, 0x40002b00, 0x00001988, 0x40002a00, 0x00002ff8,
        0x40001c00, 0x00001658, 0x00002dd8, 0x40003900, 0x40000e00, 0x00002988, 0x40003a00, 0x00003dd8, 0x40003b00, 0x00003ff8, 0x40001d00, 0x40002c00,
        0x00003988, 0x00002658, 0x40001f00, 0x40001e00, 0x80001ee0, 0x80001ec0, 0x40003c00, 0x40002d00, 0x00003658, 0x40002e00, 0x40002f00, 0x40003d00,
        0x80003da0, 0x80003d80, 0x40003e00, 0x80003ee0, 0x80003ec0, 0x00000000, 0x40003f00, 0x00001000, 0x00002000, 0x00000210, 0x00001210, 0x00003000,
        0x00000108, 0x00001108, 0x00002108, 0x00002210, 0x00003108, 0x00003210, 0xc00701c1, 0xc00709c4, 0xc00715c6, 0xc00721c9, 0xc00729cc, 0xc00735ce,
        0xc00741d1, 0xc00749d4, 0xc00755d6, 0xc00761d9, 0xc00771dd, 0xc00781e1, 0xc00791e5, 0xc007a1e9, 0xc007b1ed, 0xc007c1f1, 0xc007d1f5, 0xc007f1fd,
        0xc007e1f9, 0xc00769de, 0xc00789e6, 0xc007a9ee, 0xc007c9f6, 0xc007f9fa},
    {
        0x00000cd0, 0x40000000, 0x40001000, 0x00001cd0, 0x40002000, 0x40003000, 0x00000428, 0x40000100, 0x00000530, 0x00001428, 0x00002428, 0x00000320,
        0x40001100, 0x00003428, 0x00002cd0, 0x00001530, 0x40002300, 0x40002100, 0x00001320, 0x00000ba8, 0x00002530, 0x00003cd0, 0x40003100, 0x40001200,
        0x40001600, 0x00002320, 0x40000300, 0x40001700, 0x00001ba8, 0x00003530, 0x40001300, 0x40002600, 0x40000400, 0x40002200, 0x00003320, 0x00000ef0,
        0x40003300, 0x40001400, 0x40000600, 0x00001ef0, 0x40003200, 0x00002ba8, 0x40003800, 0x40002500, 0x40000700, 0x00002ef0, 0x40003400, 0x40002c00,
        0x00003ef0, 0x00000aa0, 0x40000800, 0x00000ff8, 0x40000900, 0x40002700, 0x40003500, 0x40000b00, 0x00001aa0, 0x40001800, 0x00002aa0, 0x40001900,
        0x40000a00, 0x40003600, 0x00000658, 0x40003700, 0x00001ff8, 0x40000c00, 0x40001b00, 0x40002800, 0x00000dd8, 0x00000778, 0x40001a00, 0x40002f00,
        0x40002900, 0x40000d00, 0x00001dd8, 0x40000f00, 0x40002b00, 0x40003d00, 0x40002a00, 0x00002ff8, 0x40001c00, 0x00001658, 0x00002dd8, 0x00001778,
        0x40003900, 0x40000e00, 0x40003a00, 0x00003dd8, 0x40003b00, 0x00003ff8, 0x40001d00, 0x00002778, 0x00002658, 0x40001f00, 0x00003778, 0x40003c00,
        0x40002d00, 0x00003658, 0x40002e00, 0x00000000, 0x40003f00, 0x00001000, 0x00002000, 0x00000210, 0x00001210, 0x00003000, 0x00000108, 0x00001108,
        0x00002108, 0x00002210, 0x00003108, 0x00003210, 0xc0080201, 0xc0080a03, 0xc0081205, 0xc0081a07, 0xc0082209, 0xc0082a0b, 0xc008320d, 0xc0083a0f,
        0xc0084211, 0xc0084a13, 0xc0086219, 0xc0086a1b, 0xc008721d, 0xc0087a1f, 0xc0088221, 0xc0088a23, 0xc0089225, 0xc0089a27, 0xc008a229, 0xc008aa2b,
        0xc008b22d, 0xc008ba2f, 0xc0090241, 0xc0090a43, 0xc009324d, 0xc0093a4f, 0xc0096259, 0xc0096a5c, 0xc009765e, 0xc0099265, 0xc0099a68, 0xc009a66a,
        0xc009b26d, 0xc009ba44, 0xc0091646, 0xc0092249, 0xc0095255, 0xc008c231, 0xc009c271, 0xc008e239, 0xc009e279, 0xc008d235, 0xc009d275, 0xc008f23d,
        0xc009f27d, 0xc0092a56, 0xc008ca72, 0xc008ea7a, 0xc008da76, 0xc008fa7e},
    {
        0x00000cd0, 0x40000000, 0x40001000, 0x00001cd0, 0x40002000, 0x40003000, 0x00000428, 0x40000100, 0x00000530, 0x00001428, 0x00002428, 0x00000320,
        0x40001100, 0x00003428, 0x00002cd0, 0x00001530, 0x40002300, 0x40002100, 0x00001320, 0x00002530, 0x00003cd0, 0x40003100, 0x40001200, 0x40001600,
        0x00002320, 0x40000300, 0x40001700, 0x00003530, 0x40001300, 0x40002600, 0x40000400, 0x40002200, 0x00003320, 0x00000ef0, 0x40003300, 0x40001400,
        0x40000600, 0x00001ef0, 0x40003200, 0x40003800, 0x40002500, 0x40000700, 0x00002ef0, 0x40003400, 0x40002c00, 0x00003ef0, 0x00000880, 0x40000800,
        0x00000ff8, 0x40000900, 0x40002700, 0x40003500, 0x40000b00, 0x00001880, 0x40001800, 0x40001900, 0x40000a00, 0x40003600, 0x00000658, 0x40003700,
        0x00001ff8, 0x00002880, 0x40000c00, 0x40001b00, 0x40002800, 0x00000dd8, 0x00000988, 0x00000778, 0x40001a00, 0x40002f00, 0x40002900, 0x40000d00,
        0x00001dd8, 0x40000f00, 0x40002b00, 0x00001988, 0x40003d00, 0x40002a00, 0x00002ff8, 0x40001c00, 0x00001658, 0x00002dd8, 0x00001778, 0x40003900,
        0x40000e00, 0x00002988, 0x40003a00, 0x00003dd8, 0x40003b00, 0x00003ff8, 0x40001d00, 0x00002778, 0x00002658, 0x40001f00, 0x00003778, 0x40003c00,
        0x40002d00, 0x00003658, 0x40002e00, 0x00000000, 0x40003f00, 0x00001000, 0x00002000, 0x00000210, 0x00001210, 0x00003000, 0x00000108, 0x00001108,
        0x00002108, 0x00002210, 0x00003108, 0x00003210, 0xc00b02c1, 0xc00b0ac3, 0xc00a0281, 0xc00a0a83, 0xc00a1285, 0xc00a1a87, 0xc00a2289, 0xc00a2a8b,
        0xc00a328d, 0xc00a3a8f, 0xc00a4291, 0xc00a4a93, 0xc00a6299, 0xc00a6a9b, 0xc00a729d, 0xc00a7a9f, 0xc00a82a1, 0xc00a8aa3, 0xc00a92a5, 0xc00a9aa7,
        0xc00aa2a9, 0xc00aaaab, 0xc00a5295, 0xc00a5a97, 0xc00ab2ad, 0xc00abaaf, 0xc00b32cd, 0xc00b3ad8, 0xc00b66da, 0xc00b72dd, 0xc00b7ae4, 0xc00b96e6,
        0xc00ba2e9, 0xc00baaec, 0xc00bb6ee, 0xc00ac2b1, 0xc00bc2f1, 0xc00ae2b9, 0xc00be2f9, 0xc00ad2b5, 0xc00bd2f5, 0xc00b42d1, 0xc00af2bd, 0xc00bf2fd,
        0xc00b82e1, 0xc00acaf2, 0xc00aeafa, 0xc00adaf6, 0xc00b4abe, 0xc00bfae2},
    {
        0x40000000, 0x40001000, 0x40002000, 0x40000200, 0x40003000, 0x00000428, 0x40000100, 0x00000530, 0x40000500, 0x00001428, 0x00002428, 0x00000320,
        0x40001100, 0x00003428, 0x00001530, 0x40001500, 0x40002100, 0x00001320, 0x00000ba8, 0x00002530, 0x40002400, 0x40003100, 0x40001200, 0x00002320,
        0x40000300, 0x00001ba8, 0x00003530, 0x40001300, 0x40000400, 0x40002200, 0x00003320, 0x00000ef0, 0x40003300, 0x40001400, 0x40000600, 0x00001ef0,
        0x40003200, 0x00002ba8, 0x40002500, 0x40000700, 0x00002ef0, 0x40003400, 0x00000aa0, 0x00000880, 0x40001e00, 0x40000800, 0x00003ba8, 0x00000ff8,
        0x40000900, 0x40002700, 0x40003500, 0x40000b00, 0x00001aa0, 0x00001880, 0x40001800, 0x00002aa0, 0x40001900, 0x40000a00, 0x40003600, 0x00000658,
        0x40003700, 0x00001ff8, 0x00002880, 0x40000c00, 0x40001b00, 0x40002800, 0x00003aa0, 0x00000988, 0x00000778, 0x40001a00, 0x00003880, 0x40002900,
        0x40000d00, 0x40000f00, 0x40002b00, 0x00001988, 0x40002a00, 0x00002ff8, 0x40001c00, 0x00001658, 0x00001778, 0x40003e00, 0x40003900, 0x40000e00,
        0x00002988, 0x40003a00, 0x40003b00, 0x40001d00, 0x00003988, 0x00002778, 0x00002658, 0x40001f00, 0x00003778, 0x40003c00, 0x40002d00, 0x00003658,
        0x40002e00, 0x00000000, 0x40003f00, 0x00001000, 0x00002000, 0x00000210, 0x00001210, 0x00003000, 0x00000108, 0x00001108, 0x00002108, 0x00002210,
        0x00003108, 0x00003210, 0xc00c2309, 0xc00c2b0b, 0xc00c330d, 0xc00c3b0f, 0xc00c4311, 0xc00c4b13, 0xc00c731d, 0xc00c7b1f, 0xc00c8321, 0xc00c8b23,
        0xc00c5315, 0xc00c5b17, 0xc00c6319, 0xc00c6b1b, 0xc00c9325, 0xc00c9b27, 0xc00ca329, 0xc00cab2b, 0xc00cb32d, 0xc00cbb2f, 0xc00d0341, 0xc00d0b43,
        0xc00d1345, 0xc00d1b47, 0xc00d2349, 0xc00d2b4b, 0xc00d334d, 0xc00d3b4f, 0xc00d4351, 0xc00d4b54, 0xc00d5756, 0xc00d6359, 0xc00d6b5c, 0xc00d775e,
        0xc00d8361, 0xc00d8b64, 0xc00d9766, 0xc00da369, 0xc00db36d, 0xc00cc331, 0xc00dc371, 0xc00cd335, 0xc00dd375, 0xc00cf33d, 0xc00df37d, 0xc00ce339,
        0xc00de379, 0xc00dab6e, 0xc00ccb72, 0xc00cdb76, 0xc00cfb7e, 0xc00ceb7a},
    {
        0x00000cd0, 0x40000000, 0x40001000, 0x00001cd0, 0x40002000, 0x40000200, 0x40003000, 0x00000428, 0x40000100, 0x00000530, 0x40000500, 0x00001428,
        0x00002428, 0x00000320, 0x40001100, 0x00003428, 0x00002cd0, 0x00001530, 0x40001500, 0x40002100, 0x00001320, 0x00000ba8, 0x00002530, 0x40002400,
        0x40003100, 0x40001200, 0x00002320, 0x40000300, 0x00001ba8, 0x00003530, 0x40001300, 0x40000400, 0x40002200, 0x00003320, 0x40003300, 0x40001400,
        0x40000600, 0x40003200, 0x00002ba8, 0x40002500, 0x40000700, 0x40003400, 0x00000aa0, 0x00000880, 0x40001e00, 0x40000800, 0x00003ba8, 0x40000900,
        0x40002700, 0x40003500, 0x40000b00, 0x00001aa0, 0x00001880, 0x40001800, 0x00002aa0, 0x40001900, 0x40000a00, 0x40003600, 0x00000658, 0x40003700,
        0x00002880, 0x40000c00, 0x40001b00, 0x40002800, 0x00000dd8, 0x00003aa0, 0x00000988, 0x00000778, 0x40001a00, 0x00003880, 0x40002900, 0x40000d00,
        0x00001dd8, 0x40000f00, 0x40002b00, 0x00001988, 0x40002a00, 0x40001c00, 0x00001658, 0x00002dd8, 0x00001778, 0x40003e00, 0x40003900, 0x40000e00,
        0x00002988, 0x40003a00, 0x40003b00, 0x40001d00, 0x00003988, 0x00002778, 0x00002658, 0x40001f00, 0x00003778, 0x40003c00, 0x40002d00, 0x00003658,
        0x40002e00, 0x00000000, 0x40003f00, 0x00001000, 0x00002000, 0x00000210, 0x00001210, 0x00003000, 0x00000108, 0x00001108, 0x00002108, 0x00002210,
        0x00003108, 0x00003210, 0xc00e2389, 0xc00e2b8b, 0xc00e338d, 0xc00e3b8f, 0xc00e4391, 0xc00e4b93, 0xc00e0381, 0xc00e0b83, 0xc00e1385, 0xc00e1b87,
        0xc00e739d, 0xc00e7b9f, 0xc00e83a1, 0xc00e8ba3, 0xc00e93a5, 0xc00e9ba7, 0xc00ea3a9, 0xc00eabab, 0xc00eb3ad, 0xc00ebbaf, 0xc00f03c1, 0xc00f0bc3,
        0xc00f13c5, 0xc00f1bc7, 0xc00f23c9, 0xc00f2bcb, 0xc00f33cd, 0xc00f3bcf, 0xc00f43d1, 0xc00f4bd4, 0xc00f57d6, 0xc00f63d9, 0xc00f6bdc, 0xc00f77de,
        0xc00f83e1, 0xc00f8be4, 0xc00f97e6, 0xc00fa3e9, 0xc00fb3ed, 0xc00ec3b1, 0xc00fc3f1, 0xc00ed3b5, 0xc00fd3f5, 0xc00ef3bd, 0xc00ff3fd, 0xc00ee3b9,
        0xc00fe3f9, 0xc00fabee, 0xc00ecbf2, 0xc00edbf6, 0xc00efbfe, 0xc00eebfa}};
static const unsigned SCHED_ITEMS[GRID][2] = {
        {0x00000000, 0x00000000}, {0x00000000, 0x00000000}, {0x00000000, 0x00000000}, {0x00000000, 0x00000000}, {0x00000000, 0x00000000}, {0x00000000, 0x00000000},
        {0x00000000, 0x00000000}, {0x00000000, 0x00000000}, {0x00000000, 0x00000000}, {0x00000000, 0x00000000}, {0x00000000, 0x00000000}, {0x00000000, 0x00000000},
        {0x00000000, 0x00000000}, {0x00000000, 0x00000000}, {0x00000000, 0x00000000}, {0x00000000, 0x00000000}, {0x00000000, 0x00000000}, {0x00000000, 0x00000000},
        {0x00000000, 0x00000000}, {0x00000000, 0x00000000}, {0x00000000, 0x00000000}, {0x00000000, 0x00000000}, {0x00000000, 0x00000000}, {0x00000000, 0x00000000},
        {0x00000000, 0x00000000}, {0x00000000, 0x00000000}, {0x00000000, 0x00000000}, {0x00000000, 0x00000000}, {0x00000000, 0x00000000}, {0x00000000, 0x00000000},
        {0x00000000, 0x00000000}, {0x00000000, 0x00000000}, {0x80026c00, 0xc0300000}, {0xa203370e, 0x00000000}, {0x80026c02, 0xc0520204}, {0xa103370c, 0xc0b14910},
        {0x80026c04, 0xc030000a}, {0xa103370a, 0xc0f36d13}, {0x80026c06, 0x80000006}, {0xa1033708, 0xc0a0481a}, {0x80026c01, 0xc0410100}, {0xa203370f, 0x00000000},
        {0x80026c03, 0xc0300005}, {0xa103370d, 0x00000000}, {0x80026c05, 0x80010103}, {0xa103370b, 0xc0a04814}, {0x80026c07, 0x80010106}, {0xa1033709, 0xc0e26c1a},
        {0xa2026c0e, 0x00000000}, {0x80011400, 0xc0520202}, {0xa1026c0c, 0xc0f36d15}, {0x80011402, 0xc0410107}, {0xa0026c0a, 0x80020203}, {0x80011404, 0xc052020b},
        {0xa0026c08, 0x80020206}, {0x80011406, 0x8002020a}, {0xa2026c0f, 0x00000000}, {0x80011401, 0xc0300003}, {0xa1026c0d, 0x00000000}, {0x80011403, 0xc0520207},
        {0xa0026c0b, 0xc0b1491f}, {0x80011405, 0xc030000c}, {0xa0026c09, 0x80000007}, {0xa0011407, 0x8000000b}, {0x80021500, 0xc0520200}, {0xa201140e, 0x00000000},
        {0x80021502, 0xc0410105}, {0xa101140c, 0x00000000}, {0x80021504, 0x80000004}, {0xa101140a, 0xc0e26c14}, {0x80021506, 0x80010107}, {0xa0011408, 0x8001010b},
        {0x80021501, 0xc0300001}, {0xa201140f, 0x00000000}, {0x80021503, 0xc0520205}, {0xa201140d, 0x00000000}, {0x80021505, 0xc041010a}, {0xa001140b, 0xc0f36d1f},
        {0xa0021507, 0x80020207}, {0xa0011409, 0x8002020b}, {0xa202150e, 0x00000000}, {0x80001300, 0xc0410103}, {0xa102150c, 0x00000000}, {0x80001302, 0xc0300008},
        {0xa102150a, 0xc0e26c10}, {0x80001304, 0xc041010c}, {0xa0021508, 0x80000008}, {0x80001306, 0x8000000c}, {0xa202150f, 0x00000000}, {0x80001301, 0xc0520203},
        {0xa202150d, 0x00000000}, {0x80001303, 0xc0410108}, {0xa002150b, 0xc0f36d10}, {0x80001305, 0xc052020c}, {0xa0021509, 0x80010108}, {0xa0001307, 0x8001010c},
        {0x80015b00, 0xc0410101}, {0xa200130e, 0x00000000}, {0x80015b02, 0xc0300006}, {0xa100130c, 0x00000000}, {0x80015b04, 0x00000000}, {0xa100130a, 0xc0a04815},
        {0x80015b06, 0xc041010e}, {0xa0001308, 0x8002020c}, {0x80015b01, 0xc0520201}, {0xa200130f, 0x00000000}, {0x80015b03, 0xc0410106}, {0xa200130d, 0x00000000},
        {0x80015b05, 0xc052020a}, {0xa000130b, 0xc0732604}, {0x80015b07, 0x80020208}, {0xa0001309, 0x8000000d}, {0xa1015b0e, 0x00000000}, {0x80027e00, 0x80020200},
        {0xa1015b0c, 0x00000000}, {0x80027e02, 0xc0520208}, {0xa0015b0a, 0x80010104}, {0x80027e04, 0xc030000d}, {0xa0015b08, 0x00000000}, {0x80027e06, 0x8001010d},
        {0xa2015b0f, 0x00000000}, {0x80027e01, 0x80000001}, {0xa2015b0d, 0x00000000}, {0x80027e03, 0xc0300009}, {0xa0015b0b, 0xc0e26c1f}, {0x80027e05, 0xc041010d},
        {0xa0015b09, 0x80000009}, {0x80027e07, 0x8002020d}, {0xa0005a00, 0x80000000}, {0xa2027e0e, 0x00000000}, {0xa0005a02, 0x80000002}, {0xa1027e0c, 0xc0b14919},
        {0xa0005a04, 0x80020204}, {0xa0027e0a, 0x80000005}, {0xa0005a06, 0x80010109}, {0xa0027e08, 0x8000000e}, {0xa0005a01, 0x80010100}, {0xa2027e0f, 0x00000000},
        {0xa0005a03, 0x80010102}, {0xa1027e0d, 0x00000000}, {0xa0005a05, 0xc0b14915}, {0xa0027e0b, 0xc0b1491c}, {0xa0005a07, 0x80020209}, {0xa0027e09, 0x8001010e},
        {0xa1005a0e, 0x00000000}, {0xa0004800, 0x80010101}, {0xa1005a0c, 0x00000000}, {0xa0004802, 0x80020202}, {0xa0005a0a, 0xc073260a}, {0xa0004804, 0x80010105},
        {0xa0005a08, 0x8000000a}, {0xa0004806, 0x8002020e}, {0xa2005a0f, 0x00000000}, {0xa0004801, 0x80020201}, {0xa2005a0d, 0x00000000}, {0xa0004803, 0x80000003},
        {0xa0005a0b, 0xc0732603}, {0xa0004805, 0x80020205}, {0xa0005a09, 0x8001010a}, {0xa0004807, 0x8000000f}, {0xa0037f00, 0xc0732606}, {0xa100480e, 0x00000000},
        {0xa0037f02, 0xc0732608}, {0xa100480c, 0x00000000}, {0xa0037f04, 0xc0b1491a}, {0xa000480a, 0xc073260b}, {0xa0037f06, 0xc073260c}, {0xa0004808, 0x8001010f},
        {0xa0037f01, 0xc073260f}, {0xa200480f, 0x00000000}, {0xa0037f03, 0xc0b14912}, {0xa200480d, 0x00000000}, {0xa0037f05, 0xc0f36d17}, {0xa000480b, 0xc0b14918},
        {0xa0037f07, 0xc0a0481e}, {0xa0004809, 0x8002020f}, {0xa2037f0e, 0x00000000}, {0xa0036d00, 0xc0732607}, {0xa1037f0c, 0xc0a04810}, {0xa0036d02, 0xc0732609},
        {0xa0037f0a, 0xc0a04811}, {0xa0036d04, 0xc0b1491b}, {0xa0037f08, 0xc0f36d1a}, {0xa0036d06, 0xc073260d}, {0xa2037f0f, 0x00000000}, {0xa0036d01, 0xc073260e},
        {0xa1037f0d, 0x00000000}, {0xa0036d03, 0xc0a0481c}, {0xa0037f0b, 0xc0e26c11}, {0xa0036d05, 0xc0f36d18}, {0xa0037f09, 0xc0e26c1e}, {0xa0036d07, 0xc0b1491e},
        {0xa1014900, 0xc0a0481d}, {0xa2036d0e, 0x00000000}, {0xa1014902, 0xc0e26c1d}, {0xa1036d0c, 0xc0f36d16}, {0xa1014904, 0xc0f36d11}, {0xa0036d0a, 0xc0e26c15},
        {0xa1014906, 0xc0732605}, {0xa0036d08, 0xc0f36d1b}, {0xa1014901, 0xc0e26c1c}, {0xa2036d0f, 0x00000000}, {0xa1014903, 0xc0b14914}, {0xa1036d0d, 0x00000000},
        {0xa1014905, 0xc0a04812}, {0xa0036d0b, 0xc0a04816}, {0xa1014907, 0xc0b14911}, {0xa0036d09, 0xc0f36d1e}, {0xa101490e, 0x00000000}, {0xa1032600, 0xc0b1491d},
        {0xa101490c, 0x00000000}, {0xa1032602, 0xc0a0481f}, {0xa101490a, 0xc0e26c12}, {0xa1032604, 0xc0e26c16}, {0xa1014908, 0xc0f36d19}, {0xa1032606, 0xc0b14913},
        {0xa201490f, 0x00000000}, {0xa1032601, 0xc0f36d1c}, {0xa201490d, 0x00000000}, {0xa1032603, 0xc0f36d1d}, {0xa101490b, 0xc0f36d12}, {0xa1032605, 0xc0a04817},
        {0xa1014909, 0xc0e26c18}, {0xa1032607, 0xc0f36d14}, {0xa1033700, 0xc0732600}, {0xa203260e, 0x00000000}, {0xa1033702, 0xc0732601}, {0xa103260c, 0xc0732602},
        {0xa1033704, 0xc0a04813}, {0xa103260a, 0xc0e26c17}, {0xa1033706, 0xc0a04819}, {0xa1032608, 0xc0a0481b}, {0xa1033701, 0xc0b14916}, {0xa103260f, 0x00000000},
        {0xa1033703, 0xc0b14917}, {0xa103260d, 0x00000000}, {0xa1033705, 0xc0e26c13}, {0xa103260b, 0xc0a04818}, {0xa1033707, 0xc0e26c19}, {0xa1032609, 0xc0e26c1b},
        {0x80011401, 0xc0300002}, {0x80011403, 0xc0300004}, {0x80011400, 0xc0520206}, {0x80021502, 0xc0410109}, {0x80026c01, 0xc030000b}, {0x80001302, 0xc052020d},
        {0x80026c00, 0xc052020e}, {0x80011402, 0xc041010f}, {0x80001300, 0xc0410102}, {0x80001304, 0xc0410104}, {0x80001301, 0xc0300007}, {0x80011404, 0xc0520209},
        {0x80021500, 0xc041010b}, {0x80001303, 0xc030000e}, {0x80021501, 0xc030000f}, {0x80021503, 0xc052020f}};
static const unsigned short SCHED_AUX[GRID][2] = {
        {0x0, 0x0}, {0x0, 0x0}, {0x0, 0x0}, {0x0, 0x0}, {0x0, 0x0}, {0x0, 0x0}, {0x0, 0x0}, {0x0, 0x0}, {0x0, 0x0}, {0x0, 0x0}, {0x0, 0x0}, {0x0, 0x0},
        {0x0, 0x0}, {0x0, 0x0}, {0x0, 0x0}, {0x0, 0x0}, {0x0, 0x0}, {0x0, 0x0}, {0x0, 0x0}, {0x0, 0x0}, {0x0, 0x0}, {0x0, 0x0}, {0x0, 0x0}, {0x0, 0x0},
        {0x0, 0x0}, {0x0, 0x0}, {0x0, 0x0}, {0x0, 0x0}, {0x0, 0x0}, {0x0, 0x0}, {0x0, 0x0}, {0x0, 0x0}, {0xe01, 0x30}, {0x0, 0x0}, {0x0, 0x30}, {0x0, 0x0},
        {0x0, 0x30}, {0x0, 0x0}, {0xe603, 0x20}, {0x0, 0x0}, {0xf01, 0x30}, {0x0, 0x0}, {0x0, 0x30}, {0x0, 0x0}, {0xe503, 0x20}, {0x0, 0x0}, {0x0, 0x20}, {0x0, 0x0},
        {0x0, 0x0}, {0x501, 0x30}, {0x0, 0x0}, {0x701, 0x30}, {0x0, 0x20}, {0x901, 0x30}, {0x0, 0x20}, {0x0, 0x20}, {0x0, 0x0}, {0x601, 0x30}, {0x0, 0x0}, {0x801, 0x30},
        {0x0, 0x0}, {0x0, 0x30}, {0x0, 0x20}, {0x0, 0x20}, {0xa01, 0x30}, {0x0, 0x0}, {0xc01, 0x30}, {0x0, 0x0}, {0x0, 0x20}, {0x0, 0x0}, {0x0, 0x20}, {0x0, 0x20},
        {0xb01, 0x30}, {0x0, 0x0}, {0xd01, 0x30}, {0x0, 0x0}, {0x0, 0x30}, {0x0, 0x0}, {0x0, 0x20}, {0x0, 0x20}, {0x0, 0x0}, {0x1, 0x30}, {0x0, 0x0}, {0x201, 0x30},
        {0x0, 0x4}, {0x401, 0x30}, {0x0, 0x20}, {0x0, 0x20}, {0x0, 0x0}, {0x101, 0x30}, {0x0, 0x0}, {0x301, 0x30}, {0x0, 0x0}, {0x0, 0x30}, {0x0, 0x20}, {0x0, 0x20},
        {0x0, 0x30}, {0x0, 0x0}, {0x0, 0x30}, {0x0, 0x0}, {0x9403, 0x0}, {0x0, 0x4}, {0x0, 0x30}, {0x0, 0x20}, {0x0, 0x30}, {0x0, 0x0}, {0x0, 0x30}, {0x0, 0x0},
        {0x0, 0x30}, {0x0, 0x0}, {0x0, 0x20}, {0x0, 0x20}, {0x0, 0x0}, {0xc003, 0x20}, {0x0, 0x0}, {0x0, 0x30}, {0x0, 0x20}, {0x0, 0x30}, {0x9803, 0x0}, {0x0, 0x20},
        {0x0, 0x0}, {0xc103, 0x20}, {0x0, 0x0}, {0x0, 0x30}, {0x0, 0x0}, {0x0, 0x30}, {0x0, 0x20}, {0x0, 0x20}, {0x0, 0x20}, {0x0, 0x0}, {0x0, 0x20}, {0x0, 0x0},
        {0x0, 0x20}, {0x0, 0x20}, {0x0, 0x20}, {0x0, 0x20}, {0x0, 0x20}, {0x0, 0x0}, {0x0, 0x20}, {0x0, 0x0}, {0x8503, 0x8}, {0x0, 0x0}, {0x0, 0x20}, {0x0, 0x20},
        {0x0, 0x0}, {0x0, 0x20}, {0x0, 0x0}, {0x0, 0x20}, {0x0, 0x0}, {0x0, 0x20}, {0x0, 0x20}, {0x0, 0x20}, {0x0, 0x0}, {0x0, 0x20}, {0x0, 0x0}, {0x0, 0x20},
        {0x0, 0x0}, {0x0, 0x20}, {0x0, 0x20}, {0x0, 0x20}, {0x0, 0x0}, {0x0, 0x0}, {0x0, 0x0}, {0x0, 0x0}, {0x0, 0x0}, {0x0, 0x0}, {0x0, 0x0}, {0x0, 0x20},
        {0x0, 0x0}, {0x0, 0x0}, {0x0, 0x8}, {0x0, 0x0}, {0x0, 0x0}, {0x0, 0x4}, {0x0, 0x0}, {0x0, 0x20}, {0x0, 0x0}, {0x0, 0x0}, {0x0, 0x0}, {0x0, 0x0},
        {0x0, 0x0}, {0x0, 0x0}, {0x0, 0x0}, {0x0, 0x0}, {0x0, 0x0}, {0x0, 0x0}, {0x0, 0x0}, {0x0, 0x0}, {0x0, 0x4}, {0x0, 0x0}, {0x0, 0x0}, {0x0, 0x0},
        {0x0, 0x0}, {0x0, 0x0}, {0xb203, 0x0}, {0x0, 0x0}, {0x0, 0x0}, {0x0, 0x8}, {0x0, 0x0}, {0x0, 0x0}, {0xb103, 0x0}, {0x0, 0x0}, {0x0, 0x4}, {0x0, 0x0},
        {0xb503, 0x0}, {0x0, 0x0}, {0x0, 0x8}, {0x0, 0x0}, {0x0, 0x0}, {0x0, 0x0}, {0x0, 0x0}, {0x0, 0x0}, {0x0, 0x0}, {0x0, 0x8}, {0x0, 0x0}, {0x0, 0x0},
        {0x0, 0x0}, {0x0, 0x0}, {0x0, 0x0}, {0x0, 0x0}, {0x0, 0x0}, {0x0, 0x0}, {0x0, 0x0}, {0x0, 0x0}, {0x0, 0x0}, {0x0, 0x0}, {0x0, 0x0}, {0x0, 0x0},
        {0x0, 0x0}, {0x0, 0x0}, {0x0, 0x0}, {0x0, 0x0}, {0x0, 0x0}, {0x0, 0x0}, {0x0, 0x0}, {0x0, 0x0}, {0x0, 0x0}, {0x0, 0x0}, {0x0, 0x0}, {0x0, 0x0},
        {0x602, 0x30}, {0x802, 0x30}, {0x502, 0x30}, {0xc02, 0x30}, {0xf02, 0x30}, {0x202, 0x30}, {0xe02, 0x30}, {0x702, 0x30}, {0x2, 0x30}, {0x402, 0x30}, {0x102, 0x30}, {0x902, 0x30},
        {0xa02, 0x30}, {0x302, 0x30}, {0xb02, 0x30}, {0xd02, 0x30}};
constexpr int NS = NC - 2;
constexpr int NE = 32;
constexpr int NQ = 32;

struct Params {
    unsigned long long x, w, out;
    unsigned long long ws[NR];
    int rank; int pad;
    unsigned head[NR][32][4];
    unsigned nhead[NR];
    unsigned quick[GRID][2];
    unsigned lbase[NR];
    unsigned nent[NR];
    unsigned ent[NR][MAXE];
    unsigned gate[NR][64];
    unsigned char posx[NR][16][4];
    unsigned short posw[NR][16][4];
    unsigned short upos[256][4];
    unsigned short epos[NR][MAXE];
    unsigned short exl[512];
    unsigned nexp, pad2;
    unsigned aux[GRID][2];
    unsigned items[GRID][2];
};
typedef const CAS Params* KP;

__device__ __forceinline__ int lds_byte(int r, int c) { const int st = (r >> 4) * 2 + (c >> 5), rr = r & 15, cc = c & 31, ob = rr * 64 + cc * 2; return st * 1024 + (ob ^ (((ob >> 9) & 1) << 5)); }
__device__ __forceinline__ void stage_rc(int b, int& R, int& C) { const int st = b / 1024, sb = b % 1024, swz = sb ^ (((sb >> 9) & 1) << 5); R = (st >> 1) * 16 + swz / 64; C = (st & 1) * 32 + (swz % 64) / 2; }
__device__ __forceinline__ int bswz(int k) { return ((k & 3) | (((k >> 3) & 1) << 2)) << 1; }
__device__ __forceinline__ unsigned pack_bf16(f32x2 v) { return __builtin_bit_cast(unsigned, __builtin_convertvector(v, bf16x2)); }
__device__ __forceinline__ f32x2 unpack_bf16(unsigned u) { return (f32x2){__uint_as_float(u << 16), __uint_as_float(u & 0xffff0000u)}; }
__device__ __forceinline__ f32x2 silu2(f32x2 y) {
    f32x2 c = {__builtin_amdgcn_fmed3f(y.x, -60.0f, 60.0f), __builtin_amdgcn_fmed3f(y.y, -60.0f, 60.0f)};
    c = c * -1.44269504089f;
    const f32x2 e = (f32x2){__builtin_amdgcn_exp2f(c.x), __builtin_amdgcn_exp2f(c.y)} + 1.0f;
    return y * (f32x2){__builtin_amdgcn_rcpf(e.x), __builtin_amdgcn_rcpf(e.y)};
}
__host__ __device__ __forceinline__ unsigned image_src(bool isw, unsigned pi, unsigned lane) {
    const unsigned kt = pi >> 5, h = (pi >> 4) & 1, st = pi & 15;
    const unsigned ox = ((st >> 1) * 16 + h * 128 + (lane >> 2)) * XP + kt * 128 + (st & 1) * 64 + (((lane & 3) * 16) ^ ((lane >> 5) << 5));
    const unsigned ow = (kt * 64 + st * 4 + (lane >> 4)) * WP + h * 256 + (((lane & 15) ^ (((lane >> 4) | (((st >> 1) & 1) << 2)) << 1)) * 16);
    return isw ? ow : ox;
}
#define RSRC(p, n) __builtin_amdgcn_make_buffer_rsrc((void*)(p), 0, (n), 0x00020000)
#define WAIT_V(n) asm volatile("s_waitcnt vmcnt(" #n ")" ::: "memory")
#define WAIT_L(n) asm volatile("s_waitcnt lgkmcnt(" #n ")" ::: "memory")
#define DRAIN_V do { asm volatile("" ::: "memory"); __builtin_amdgcn_s_waitcnt(0x0F70); asm volatile("s_waitcnt vmcnt(0)" ::: "memory"); } while (0)
#define BAR __builtin_amdgcn_s_barrier()
#define SCHED __builtin_amdgcn_sched_barrier(0)
#define CFENCE asm volatile("" ::: "memory")
#define NAN4 ((u32x4){0x7fc07fc0u, 0x7fc07fc0u, 0x7fc07fc0u, 0x7fc07fc0u})


__device__ __forceinline__ bool timed_out(gchar* ws0, unsigned call) {
    return (unsigned)__builtin_amdgcn_readfirstlane(__hip_atomic_load((gu32*)(ws0 + TMO_OFF), __ATOMIC_RELAXED, __HIP_MEMORY_SCOPE_AGENT)) == call;
}
__device__ __forceinline__ bool spin_expired(gchar* ws0, unsigned call, unsigned long long t0, unsigned& spins) {
    CFENCE;
    if (((++spins & 15) == 0 && timed_out(ws0, call)) || __builtin_amdgcn_s_memrealtime() - t0 > DEADLINE_TICKS) {
        __hip_atomic_store((gu32*)(ws0 + TMO_OFF), call, __ATOMIC_RELAXED, __HIP_MEMORY_SCOPE_AGENT);
        return true;
    }
    return false;
}
__device__ __forceinline__ void nap(int gap) {
    int n = min((max(gap - 2 * NC, 0) * CU_) >> 5, 48);
    do { __builtin_amdgcn_s_sleep(8); } while (n--);
}
__device__ __forceinline__ unsigned next_call(gu32* ctr, unsigned prev) {
    const unsigned call = (unsigned)__builtin_amdgcn_readfirstlane(prev) + 1u;
    if ((threadIdx.x & 63) == 0) __hip_atomic_store(ctr, call, __ATOMIC_RELAXED, __HIP_MEMORY_SCOPE_AGENT);
    return call;
}
__device__ __forceinline__ unsigned link_progress(const gchar* ws0, unsigned link) {
    return __builtin_amdgcn_raw_buffer_load_b32(RSRC(ws0, WS_NEED), SUM_OFF + (blockIdx.x & 7) * LS + link * 8, 0, 16);
}

typedef unsigned u32x2 __attribute__((ext_vector_type(2)));
#define KLOAD32(off) __builtin_amdgcn_raw_buffer_load_b32(RSRC(p, sizeof(Params)), (off), 0, 0)
__device__ __forceinline__ void relay_out(KP p, gchar* ws0, unsigned call, int d, int lane) {
    const auto rl = RSRC(ws0, WS_NEED), rq = RSRC(p->ws[d], WS_NEED);
    const unsigned base = call * CALL_STRIDE;
    const int total = p->nent[d] * BPC, nhead = p->nhead[d], idle = d == 1 ? p->lbase[2] * BPC : 0;
    const unsigned gate = p->gate[d][lane];
    const unsigned glink = (gate >> 9) & 7, garg = (gate >> 16) & 255;
    const unsigned goff = gate == 0xffffffffu ? OOB : glink ? SUM_OFF + (blockIdx.x & 7) * LS + glink * 8 : TFLAG_OFF + garg * 4;
    const unsigned gwant = gate == 0xffffffffu ? 0 : glink ? base + garg * BPC : call;
    unsigned eoff[3], ewant[3];
#pragma unroll
    for (int k = 0; k < 3; ++k) { const unsigned e = p->epos[d][k * 64 + lane];
        eoff[k] = !e ? OOB : e <= NQ ? EQ_OFF + (e - 1) * 4 : EPROG_OFF + ((e - 1 - NQ) % NE) * 4; ewant[k] = !e ? 0 : e <= NQ ? 4 * call : base + e; }
    const unsigned poff = PLOC_OFF + d * LS + (lane < NS ? lane : 0) * 4;
    const unsigned long long t0 = __builtin_amdgcn_s_memrealtime(); unsigned spins = 0;
    int lim = 0; bool marked = !nhead, exported = false, send = false, direct = false;
    for (;;) {
        const unsigned g = __builtin_amdgcn_raw_buffer_load_b32(rl, goff, 0, 16);
        const unsigned e0 = __builtin_amdgcn_raw_buffer_load_b32(rl, exported ? OOB : eoff[0], 0, 16);
        const unsigned e1 = __builtin_amdgcn_raw_buffer_load_b32(rl, exported ? OOB : eoff[1], 0, 16);
        const unsigned e2 = __builtin_amdgcn_raw_buffer_load_b32(rl, exported ? OOB : eoff[2], 0, 16);
        const unsigned pl = __builtin_amdgcn_raw_buffer_load_b32(rl, marked ? OOB : poff, 0, 16);
        CFENCE;
        __builtin_amdgcn_raw_buffer_store_b32(base + lim, rl, (send && lane == 0) ? RMSG_OFF + d * LS : OOB, 0, 17);
        __builtin_amdgcn_raw_buffer_store_b32(base + lim, rq, (direct && lane < 8) ? LIM_OFF + lane * LS + d * 8 + 4 : OOB, 0, 17);
        CFENCE;
        if (lim == total || spin_expired(ws0, call, t0, spins)) break;
        const unsigned long long ng = __ballot(g < gwant);
        int nl = ng ? (__builtin_amdgcn_readlane((int)gate, __builtin_ctzll(ng)) & 511) * BPU : total;
        if (!exported) {
            const unsigned long long n0 = __ballot(e0 < ewant[0]), n1 = __ballot(e1 < ewant[1]), n2 = __ballot(e2 < ewant[2]);
            exported = !(n0 | n1 | n2);
            if (!exported) nl = min(nl, (n0 ? __builtin_ctzll(n0) : n1 ? 64 + __builtin_ctzll(n1) : 128 + __builtin_ctzll(n2)) * BPC);
        }
        marked = marked || !__ballot(pl < base);
        if (!marked) nl = 0;
        send = nl > lim;
        direct = send && (lim < nhead + NS || (unsigned)(lim - idle) < NS);
        lim = max(lim, nl);
        if (!send) __builtin_amdgcn_s_sleep(2);
    }
    DRAIN_V;
}
__device__ __forceinline__ void relay_limit(KP p, gchar* ws0, unsigned call, int d, int lane) {
    const auto rl = RSRC(ws0, WS_NEED), rq = RSRC(p->ws[d], WS_NEED);
    const unsigned fin = call * CALL_STRIDE + p->nent[d] * BPC;
    const unsigned long long t0 = __builtin_amdgcn_s_memrealtime(); unsigned spins = 0, val = 0; bool up = false;
    for (;;) {
        const unsigned m = __builtin_amdgcn_raw_buffer_load_b32(rq, RMSG_OFF + d * LS, 0, 17);
        CFENCE;
        __builtin_amdgcn_raw_buffer_store_b32(val, rl, (up && lane < 8) ? LIM_OFF + lane * LS + d * 8 : OOB, 0, 16);
        CFENCE;
        if (val >= fin || spin_expired(ws0, call, t0, spins)) break;
        const unsigned mx = __builtin_amdgcn_readfirstlane(m);
        up = mx > val;
        val = max(val, mx);
        if (!up) __builtin_amdgcn_s_sleep(2);
    }
    DRAIN_V;
}
__device__ __forceinline__ void relay_arrival(KP p, gchar* ws0, unsigned call, int lane, bool mine) {
    const unsigned nhead = p->nhead[lane >> 3];
    const auto rl = RSRC(ws0, WS_NEED);
    const unsigned long long t0 = __builtin_amdgcn_s_memrealtime(); unsigned spins = 0, val = 0, psh = lane < 16 ? call : 0; bool up = false;
    for (;;) {
        const u32x4 v = __builtin_amdgcn_raw_buffer_load_b128(rl, PLOC_OFF + (lane >> 3) * LS + (lane & 7) * 16, 0, 16);
        const u32x4 q = __builtin_amdgcn_raw_buffer_load_b128(rl, psh >= call ? OOB : HEAD_OFF + (lane >> 3) * LS + (lane & 7) * 16, 0, 17);
        CFENCE;
        __builtin_amdgcn_raw_buffer_store_b32(val, rl, up ? SUM_OFF + (lane & 7) * LS + (lane >> 3) * 8 : OOB, 0, 16);
        CFENCE;
        if (!__ballot(lane >= 8 && val < call * CALL_STRIDE + CALL_STRIDE - 1) || spin_expired(ws0, call, t0, spins)) break;
        const int w0 = (lane & 7) * 4;
        unsigned a = min(min(w0 < NS ? v.x : ~0u, w0 + 1 < NS ? v.y : ~0u), min(w0 + 2 < NS ? v.z : ~0u, w0 + 3 < NS ? v.w : ~0u));
        a = min(a, (unsigned)__shfl_xor((int)a, 1)); a = min(a, (unsigned)__shfl_xor((int)a, 2)); a = min(a, (unsigned)__shfl_xor((int)a, 4));
        if (__ballot(psh < call)) {
            unsigned b = min(min(w0 < NS ? q.x : ~0u, w0 + 1 < NS ? q.y : ~0u), min(w0 + 2 < NS ? q.z : ~0u, w0 + 3 < NS ? q.w : ~0u));
            b = min(b, (unsigned)__shfl_xor((int)b, 1)); b = min(b, (unsigned)__shfl_xor((int)b, 2)); b = min(b, (unsigned)__shfl_xor((int)b, 4));
            psh = max(psh, b);
        }
        a = max(a, call * CALL_STRIDE + nhead);
        if (psh < call) a = 0;
        up = mine && a > val;
        val = max(val, a);
    }
    DRAIN_V;
}

struct CopySlot { unsigned src, dst; };
template <int N> __device__ __forceinline__ void wait_vm() { asm volatile("s_waitcnt vmcnt(%0)" :: "i"(N) : "memory"); }
__device__ __forceinline__ CopySlot copy_decode(unsigned e, int g, unsigned slot0, unsigned par, int lane) {
    const unsigned type = e >> 30, sub = g % BPC;
    const unsigned ry = ((e >> (sub < BPU ? 10 : 0)) & 1023) * 32768 + (sub % BPU) * (CU_ * 1024);
    const unsigned so = type == 3 ? YOUT_REL + ry : (type == 2 ? 0 : XEX_REL) + ((e >> 16) & 1023) * 65536 + sub * (CU_ * 1024);
    const unsigned dn = type == 3 ? YIN_OFF + ry : par + slot0 * 65536 + g * (CU_ * 1024);
    CopySlot s; s.src = so + lane * 16; s.dst = dn + lane * 16;
    return s;
}
__device__ __forceinline__ unsigned pick_entry(int e0, int e1, int e2, int en) {
    const unsigned ea = __builtin_amdgcn_readlane(e0, en & 63), eb = __builtin_amdgcn_readlane(e1, en & 63), ec = __builtin_amdgcn_readlane(e2, en & 63);
    return en >= 128 ? ec : en >= 64 ? eb : ea;
}
__device__ __forceinline__ unsigned copy_grab(gu32* cnt, int lane) { unsigned r = 0; if (lane == 0) r = __hip_atomic_fetch_add(cnt, 1u, __ATOMIC_RELAXED, __HIP_MEMORY_SCOPE_AGENT); return r; }

struct Pre { unsigned long long x, w, wsd; unsigned slot0, nent, nhead, he[NPUSH]; };
__device__ __forceinline__ void copy_stream(KP p, gchar* ws0, gu32* ctr, unsigned prev, int d, int lane, const Pre pre) {
    static_assert(CU_ == 4, "hand-counted waits");
    const unsigned long long px = pre.x, pw = pre.w, wsd = pre.wsd;
    const unsigned slot0 = pre.slot0, nent = pre.nent, nhead = pre.nhead;
    unsigned he[NPUSH];
#pragma unroll
    for (int j = 0; j < NPUSH; ++j) he[j] = pre.he[j];
    u32x4 h[NPUSH][CU_];
    const unsigned lx = image_src(false, 0, lane), lw0 = image_src(true, 0, lane), lw1 = image_src(true, 2, lane);
#pragma unroll
    for (int j = 0; j < NPUSH; ++j) {
        const unsigned e = he[j], isw = e & 1;
        const auto rs = RSRC(isw ? pw : px, e != ~0u ? X_BYTES : 0);
#pragma unroll
        for (int u = 0; u < CU_; ++u)
            h[j][u] = __builtin_amdgcn_raw_buffer_load_b128(rs, isw ? (u < 2 ? lw0 : lw1) : lx, (e & 0xfffffeu) + (isw ? (u & 1) * 4 * WP : (u >> 1) * 16 * XP + (u & 1) * 64), 0);
        CFENCE;
    }
    const int e0 = KLOAD32(offsetof(Params, ent) + (d * MAXE + lane) * 4);
    const int e1 = KLOAD32(offsetof(Params, ent) + (d * MAXE + 64 + lane) * 4);
    const int e2 = KLOAD32(offsetof(Params, ent) + (d * MAXE + 128 + lane) * 4);
    CFENCE;
    const unsigned call = next_call(ctr, prev);
    const unsigned par = XST_OFF + (call & 1) * PAR_BYTES;
    const auto rq = RSRC(wsd + par, PAR_BYTES), rl = RSRC(ws0, WS_NEED);
    const unsigned poff = lane == 0 ? PLOC_OFF + d * LS + blockIdx.x * 4 : OOB, loff = LIM_OFF + (blockIdx.x & 7) * LS + d * 8;
    int vzero = 0; asm volatile("" : "+v"(vzero));
    gu32* cnt = (gu32*)(ws0 + CNT_OFF + ((call & 1) * NR + d) * LS) + vzero;
    const int total = nent * BPC;
    const unsigned base = call * CALL_STRIDE;
    int limit = 0;
    if (nhead) {
        const bool late = he[0] == ~0u;
        u32x2 pc = {0, 0};
#pragma unroll
        for (int k = 0; k < NPUSH * CU_; ++k) {
            constexpr int MK = NPUSH * CU_ - MARK_LEAD, PK = NPUSH * CU_ - 3;
            if (k < CU_ && late) continue;
            if (k < 2 || (k < CU_ + 2 && late)) {}
            else if ((k - 1 == MK) + (k - 1 == PK) + (k - 2 == MK) + (k - 2 == PK) == 2) wait_vm<3>();
            else if (k - 1 == MK || k - 1 == PK || k - 2 == MK || k - 2 == PK) wait_vm<2>(); else wait_vm<1>();
            __builtin_amdgcn_raw_buffer_store_b128(h[k / CU_][k % CU_], rq, slot0 * 65536 + (he[k / CU_] >> 24) * (CU_ * 1024) + (k % CU_) * 1024 + lane * 16, 0, 17); CFENCE;
            if (k == MK) { __builtin_amdgcn_raw_buffer_store_b32(base, rl, poff, 0, 16); CFENCE; }
            if (k == PK) { pc = __builtin_amdgcn_raw_buffer_load_b64(rl, loff, 0, 17); CFENCE; }
        }
        limit = min(total, max((int)(__builtin_amdgcn_readfirstlane(max(pc.x, pc.y)) - base), 0));
    }
#define PUBLISH(v) __builtin_amdgcn_raw_buffer_store_b32(base + (v), rl, poff, 0, 16)
#define LD1(sl, u) do { t[u] = __builtin_amdgcn_raw_buffer_load_b128(rq, sl.src + (u) * 1024, 0, 17); CFENCE; } while (0)
#define ST1(sl, u) do { __builtin_amdgcn_raw_buffer_store_b128(t[u], rl, sl.dst + (u) * 1024, 0, 17); CFENCE; } while (0)
    const auto rh = RSRC(wsd, WS_NEED);
    unsigned hoff = (nhead && lane == 0) ? HEAD_OFF + d * LS + blockIdx.x * 4 : OOB;
    int cur = nhead + blockIdx.x;
    while (cur < total) {
        if (cur >= limit) {
            DRAIN_V;
            __builtin_amdgcn_raw_buffer_store_b32(call, rh, hoff, 0, 17); hoff = OOB;
            PUBLISH(cur);
            const unsigned long long t0 = __builtin_amdgcn_s_memrealtime(); unsigned spins = 0;
            for (;;) {
                const u32x2 q = __builtin_amdgcn_raw_buffer_load_b64(rl, loff, 0, 17);
                limit = min(total, max((int)(__builtin_amdgcn_readfirstlane(max(q.x, q.y)) - base), 0));
                if (cur < limit) break;
                if (spin_expired(ws0, call, t0, spins)) return;
                __builtin_amdgcn_s_sleep(8);
            }
            CFENCE;
        }
        u32x4 t[CU_];
        CopySlot s = copy_decode(pick_entry(e0, e1, e2, cur / BPC), cur, slot0, par, lane);
        wait_vm<1>(); LD1(s, 0); __builtin_amdgcn_raw_buffer_store_b32(0, rl, OOB, 0, 0); CFENCE;
        wait_vm<2>(); LD1(s, 1); __builtin_amdgcn_raw_buffer_store_b32(call, rh, hoff, 0, 17); hoff = OOB; CFENCE;
        do {
            const bool near = limit < total && cur + 4 * NC >= limit;
            CFENCE; const unsigned drawn = copy_grab(cnt, lane); CFENCE;
            const unsigned pc = __builtin_amdgcn_raw_buffer_load_b32(rl, near ? loff : OOB, 0, 16); CFENCE;
            asm volatile("" : "+v"(t[0])); LD1(s, 2); ST1(s, 0);
            wait_vm<4>();
            PUBLISH(cur); CFENCE;
            LD1(s, 3); ST1(s, 1);
            asm volatile("" : "+v"(t[2]));
            cur = __builtin_amdgcn_readfirstlane(drawn) + nhead + NS;
            limit = min(total, max(limit, near ? (int)(__builtin_amdgcn_readfirstlane(pc) - base) : 0));
            CopySlot n = copy_decode(pick_entry(e0, e1, e2, cur / BPC), cur, slot0, par, lane);
            if (cur >= limit) n.src = OOB;
            LD1(n, 0); ST1(s, 2);
            asm volatile("" : "+v"(t[3])); LD1(n, 1); ST1(s, 3);
            s = n;
        } while (cur < limit);
    }
    DRAIN_V;
    PUBLISH(CALL_STRIDE - 1);
#undef LD1
#undef ST1
#undef PUBLISH
}

__device__ __forceinline__ void copy_role(KP p, gchar* ws0, gu32* ctr, unsigned prev, int d, const Pre pre) {
    const int wid = __builtin_amdgcn_readfirstlane(threadIdx.x >> 6), lane = threadIdx.x & 63;
    if (blockIdx.x < NS && wid < NR - 1) {
        copy_stream(p, ws0, ctr, prev, d, lane, pre);
        return;
    }
    const unsigned call = next_call(ctr, prev);
    if (blockIdx.x == NC - 1 && wid < NR - 1) relay_out(p, ws0, call, wid + 1, lane);
    else if (wid < NR - 1) {}
    else if (blockIdx.x >= NC - 2) relay_arrival(p, ws0, call, lane, ((lane ^ blockIdx.x) & 1) != 0);
    else {
        if (blockIdx.x >= 1 && blockIdx.x < NR) relay_limit(p, ws0, call, blockIdx.x, lane);
        if (blockIdx.x == 0 && lane < NR)
            __hip_atomic_store((gu32*)(ws0 + CNT_OFF + (((call + 1) & 1) * NR + lane) * LS), 0u, __ATOMIC_RELAXED, __HIP_MEMORY_SCOPE_AGENT);
    }
}

__device__ __forceinline__ unsigned export_quick(gchar* ws0, gu32* ctr, unsigned prev, unsigned qi, unsigned e, unsigned long long px, unsigned long long pw) {
    const int wid = __builtin_amdgcn_readfirstlane(threadIdx.x >> 6), lane = threadIdx.x & 63;
    const unsigned isw = e & 1;
    const auto rs = RSRC(isw ? pw : px, X_BYTES);
    const u32x4 a = __builtin_amdgcn_raw_buffer_load_b128(rs, image_src(isw, wid * 2, lane), e & ~1u, 0), b = __builtin_amdgcn_raw_buffer_load_b128(rs, image_src(isw, wid * 2 + 1, lane), e & ~1u, 0);
    CFENCE;
    const unsigned call = next_call(ctr, prev);
    const auto rd = RSRC(ws0 + (XST_OFF + (call & 1) * PAR_BYTES + XEX_REL), 2 * X_BYTES);
    __builtin_amdgcn_raw_buffer_store_b128(a, rd, qi * 16384 + wid * 2048 + lane * 16, 0, 17);
    __builtin_amdgcn_raw_buffer_store_b128(b, rd, qi * 16384 + wid * 2048 + 1024 + lane * 16, 0, 17);
    DRAIN_V; __syncthreads();
    if (threadIdx.x == 0) __hip_atomic_fetch_add((gu32*)(ws0 + EQ_OFF) + (qi >> 2), 1u, __ATOMIC_RELAXED, __HIP_MEMORY_SCOPE_AGENT);
    return call;
}
__device__ __forceinline__ unsigned export_role(KP p, gchar* ws0, gu32* ctr, unsigned prev, int slot, unsigned e1st, unsigned rank, unsigned long long px, unsigned long long pw) {
    const int idx = slot + NQ;
    const int wid = __builtin_amdgcn_readfirstlane(threadIdx.x >> 6), lane = threadIdx.x & 63;
    const int n = e1st >> 16; e1st &= 0xffff;
    const int el = __builtin_amdgcn_raw_buffer_load_b16(RSRC(p, sizeof(Params)), offsetof(Params, exl) + ((idx + (lane & 31) * NE) & 511) * 2, 0, 0);
    u32x4 a[8], b[8];
#define ELOAD(t, k) do { const unsigned _e = (k) ? __builtin_amdgcn_readlane(el, (k) & 31) : e1st, _w = _e >> 8, _ks = (_e >> 5) & 7, _pn = _e & 31; \
        const unsigned _c = _w ? _ks * 128 * WP + (((rank >> 2) * 16) ^ _pn) * 512 : ((rank ^ (_pn >> 2)) * 4 + (_pn & 3)) * 256 * XP + _ks * 256; \
        const auto _rs = RSRC(_w ? pw : px, idx + (k) * NE < n ? X_BYTES : 0); \
        _Pragma("unroll") for (int u = 0; u < 8; ++u) t[u] = __builtin_amdgcn_raw_buffer_load_b128(_rs, image_src(_w, wid * 8 + u, lane), _c, 0); } while (0)
#define ESTEP(t, tn, k) { if (idx + (k) * NE >= n) break; \
        ELOAD(tn, (k) + 1); CFENCE; \
        _Pragma("unroll") for (int u = 0; u < 8; ++u) __builtin_amdgcn_raw_buffer_store_b128(t[u], rd, (idx + (k) * NE) * 65536 + wid * 8192 + lane * 16 + u * 1024, 0, 17); \
        DRAIN_V; __syncthreads(); \
        if (threadIdx.x == 0) __hip_atomic_store((gu32*)(ws0 + EPROG_OFF) + slot, call * CALL_STRIDE + idx + (k) * NE + 1, __ATOMIC_RELAXED, __HIP_MEMORY_SCOPE_AGENT); }
    CFENCE; ELOAD(a, 0); CFENCE;
    const unsigned call = next_call(ctr, prev);
    const auto rd = RSRC(ws0 + (XST_OFF + (call & 1) * PAR_BYTES + XEX_REL), 2 * X_BYTES);
    for (int k = 0;; k += 2) { ESTEP(a, b, k) ESTEP(b, a, k + 1) }
#undef ELOAD
#undef ESTEP
    DRAIN_V;
    return call;
}

__device__ __forceinline__ int wait_steps(gchar* ws0, LAS unsigned char* lds, int want, unsigned call, int wid, int lane) {
    LAS unsigned* ctl = (LAS unsigned*)(lds + LDS_CTL);
    if (wid == 0) {
        const unsigned nx = ctl[64 + lane], nw = ctl[128 + lane], info = ctl[192 + lane];
        const unsigned long long t0 = __builtin_amdgcn_s_memrealtime(); unsigned spins = 0;
        int upto;
        for (;;) {
            const unsigned mx = link_progress(ws0, info & 7), mw = link_progress(ws0, (info >> 8) & 7);
            const unsigned long long nok = __ballot(mx < nx || mw < nw);
            upto = nok ? __builtin_ctzll(nok) : 64;
            if (upto > want) break;
            if (spin_expired(ws0, call, t0, spins)) { upto = 64; break; }
            nap(__builtin_amdgcn_readlane(max((int)(nx - mx), (int)(nw - mw)), upto));
        }
        if (lane == 0) ctl[0] = (unsigned)upto;
        asm volatile("s_waitcnt vmcnt(0) lgkmcnt(0)" ::: "memory");
    }
    CFENCE; BAR; CFENCE;
    const int r = __builtin_amdgcn_readfirstlane(ctl[0]);
    CFENCE;
    return r;
}
__device__ __forceinline__ int wait_quarters(gchar* ws0, LAS unsigned char* lds, int want, int word, unsigned call, int wid, int lane) {
    LAS unsigned* ctl = (LAS unsigned*)(lds + LDS_CTL);
    if (wid == 0) {
        const unsigned need = ctl[64 + lane], info = ctl[192 + lane];
        const unsigned off = lane == 8 ? TMO_OFF : lane > 8 ? OOB : (info & 7) ? SUM_OFF + (blockIdx.x & 7) * LS + (info & 7) * 8 : TFLAG_OFF + ((info >> 8) & 511) * 4;
        const unsigned long long t0 = __builtin_amdgcn_s_memrealtime(); unsigned spins = 0;
        int known;
        for (;;) {
            const unsigned mine = __builtin_amdgcn_raw_buffer_load_b32(RSRC(ws0, WS_NEED), off, 0, 16);
            const unsigned ok = (unsigned)__ballot(mine >= need);
            known = ok & (ok >> 4) & 15;
            if ((unsigned)__builtin_amdgcn_readlane((int)mine, 8) == call) known = 31;
            if (known & want) break;
            if (spin_expired(ws0, call, t0, spins)) { known = 31; break; }
            const int gap = min((int)(need - mine), (int)ctl[128 + lane] - (int)mine), f = __builtin_ctz(want), l = 31 - __builtin_clz(want);
            nap(min(max(__builtin_amdgcn_readlane(gap, f), __builtin_amdgcn_readlane(gap, f + 4)), max(__builtin_amdgcn_readlane(gap, l), __builtin_amdgcn_readlane(gap, l + 4))));
        }
        if (spins) known |= 32;
        if (lane == 0) ctl[1 + word] = (unsigned)known;
        WAIT_L(0);
    }
    CFENCE; BAR; CFENCE;
    const int r = __builtin_amdgcn_readfirstlane(ctl[1 + word]);
    CFENCE;
    return r;
}

__device__ __forceinline__ void gemm_role(KP p, gchar* ws0, LAS unsigned char* lds, unsigned call) {
    const int tid = threadIdx.x, wid = __builtin_amdgcn_readfirstlane(tid >> 6), lane = tid & 63, wr = wid >> 2, wc = wid & 3, fr = lane & 15, fq = lane >> 4;
    const unsigned rank = p->rank;
    const unsigned long long px = p->x, pw = p->w, stg = (unsigned long long)ws0 + (XST_OFF + (call & 1) * PAR_BYTES);
    LAS unsigned* ctl = (LAS unsigned*)(lds + LDS_CTL);
    unsigned voffA, voffB;
    { int R, C; stage_rc(tid * 16, R, C); voffA = (unsigned)R * XP + C * 2;
      const int k = wid * 4 + (lane >> 4); voffB = (unsigned)k * WP + (((lane & 15) ^ bswz(k)) * 16); }
    const unsigned ldsw = (unsigned)wid * 1024u;
    const int aoff = lds_byte(wr * 64 + fr, fq * 8);
    unsigned boff[2];
    { const int k = 8 * fq + ((lane >> 2) & 3), pp = lane & 3;
#pragma unroll
      for (int n = 0; n < 2; ++n) boff[n] = (unsigned)(size_t)lds + 4 * HTB + k * 256 + (((wc * 4 + n * 2 + (pp >> 1)) ^ bswz(k)) * 16) + (pp & 1) * 8; }
    const unsigned vlin = (unsigned)tid * 16;
#define SA(b, h) (((b) * 2 + (h)) * HTB)
#define SB(b, h) ((4 + (b) * 2 + (h)) * HTB)
#define STAGE(bufoff, rs, voff, so, pstep) do { _Pragma("unroll") for (int _i = 0; _i < 2; ++_i) \
        __builtin_amdgcn_raw_ptr_buffer_load_lds(rs, (LAS void*)(lds + (bufoff) + ldsw + _i * 8192), 16, voff, (so) + _i * (pstep), 0, 0); } while (0)
#define STA(b, h, st, o) STAGE(SA(b, h), RSRC((st) ? stg : px, (st) ? STG_BYTES : X_BYTES), (st) ? vlin : voffA, (o) + ((st) ? ((b) * 2 + (h)) * HTB : (b) * 128 + (h) * 128 * XP), (st) ? 8192u : 64 * XP)
#define STB(b, h, st, o) STAGE(SB(b, h), RSRC((st) ? stg : pw, (st) ? STG_BYTES : X_BYTES), (st) ? vlin : voffB, (o) + ((st) ? ((b) * 2 + (h)) * HTB : (b) * 64 * WP + (h) * 256), (st) ? 8192u : 32 * WP)
#define LDA(dst, b, h) do { _Pragma("unroll") for (int m = 0; m < 4; ++m) _Pragma("unroll") for (int k = 0; k < 2; ++k) dst[m][k] = *(const LAS bf16x8*)(lds + SA(b, h) + aoff + m * 2048 + k * 1024); } while (0)
#define TRRD(dst, addr, off) asm volatile("ds_read_b64_tr_b16 %0, %1 offset:%2" : "=&v"(dst) : "v"(addr), "i"(off) : "memory")
#define LDB(dst, b, h) do { _Pragma("unroll") for (int n = 0; n < 2; ++n) _Pragma("unroll") for (int k = 0; k < 2; ++k) { s16x4 _lo, _hi; \
        TRRD(_lo, boff[n], ((b) * 2 + (h)) * HTB + k * 8192); TRRD(_hi, boff[n], ((b) * 2 + (h)) * HTB + k * 8192 + 1024); \
        dst[n][k] = __builtin_shufflevector(_lo, _hi, 0, 1, 2, 3, 4, 5, 6, 7); } } while (0)
#define MMA(ai, bj, At, Bt) do { __builtin_amdgcn_s_setprio(1); _Pragma("unroll") for (int m = 0; m < 4; ++m) _Pragma("unroll") for (int n = 0; n < 2; ++n) _Pragma("unroll") for (int k = 0; k < 2; ++k) \
        acc[ai][bj][m][n] = __builtin_amdgcn_mfma_f32_16x16x32_bf16(Bt[n][k], At[m][k], acc[ai][bj][m][n], 0, 0, 0); __builtin_amdgcn_s_setprio(0); } while (0)
#define REMOTE(a, b, s) do { const unsigned _i = __builtin_amdgcn_readlane(steps, s); a = ((_i >> 12) & 1023) << 16; b = (_i >> 22) << 16; } while (0)

    for (int it = 0; it < 2; ++it) {
        const unsigned item = p->items[blockIdx.x][it];
        if (!(item >> 31)) break;
        const bool fin = item & 0x40000000u;
        const unsigned cp = item & 31, rp = (item >> 8) & 15, dq = (item >> 12) & 7, op = (item >> 16) & 3;
        const unsigned aux = p->aux[blockIdx.x][it], mode = aux & 3;
        const bool half = !fin && (mode == 1 || mode == 2);
        const int nown = half ? (mode == 1 ? OWN_MAIN : 8 - OWN_MAIN) : (fin || dq != 0) ? 8 : 0, nrem = dq >= 4 ? 19 : dq >= 2 ? 20 : 24;
        const int nsteps = (fin && !(aux & 16)) ? (cp >> 4) * 2 + 8 : nown + ((half || (aux & 32)) ? (nrem + (mode == 2)) / 2 : nrem);
        const unsigned row0 = (fin ? rank : rank ^ dq) * 1024 + op * 256;
        const unsigned colb = ((rank >> 2) * 16 ^ cp) * 512;
        if (it) __syncthreads();
        unsigned cA = row0 * XP + (mode == 2 ? OWN_MAIN * 256 : 0), cB = colb + (mode == 2 ? OWN_MAIN * 128 * WP : 0); bool cs = !nown;
        if (nown) { STB(0, 0, false, cB); STA(0, 0, false, cA); STB(0, 1, false, cB); STA(0, 1, false, cA); }
        if (wid == 0 && nsteps > nown) {
            int ln = lane; asm volatile("" : "+v"(ln));
            unsigned nx = 0, nw = 0, info = 0; int slot = ln;
            {
                const bool far = cp >= 16;
                const int round = far ? ln : ln / 6, dd = far ? 1 : 2 + ln - round * 6;
                bool ok = far ? ln < 2 : ln < 24 && dd != (int)dq && !(dq >= 4 && dd == (int)(dq ^ 1) && round == XR);
                const unsigned ex = p->posx[dd][far ? op : rp][round & 3], ew = p->posw[dd][cp & 15][round & 3];
                const int key = ok ? (int)(max(ex, ew & 255) * 64 + ln) : 0x7fff0000 + ln;
                int rk = 0;
                for (int j = 0; j < 24; ++j) rk += __builtin_amdgcn_readlane(key, j) < key;
                slot = rk + nown;
                if (half || (aux & 32)) { ok = ok && (mode == 1 || fin) == (rk & 1); slot = nown + (rk >> 1); }
                ctl[64 + ln] = 0; ctl[128 + ln] = 0; ctl[192 + ln] = 0;
                if (!ok) slot = 63;
                if (ok) { nx = call * CALL_STRIDE + (ex + 1) * BPC; nw = call * CALL_STRIDE + ((ew & 255) + 1) * BPC; info = dd | (ew & 0x700) | ((p->lbase[dd] + ex) << 12) | ((p->lbase[ew >> 8] + (ew & 255)) << 22); }
            }
            ctl[64 + slot] = nx; ctl[128 + slot] = nw; ctl[192 + slot] = info;
            WAIT_L(0);
        }
        int upto = nown, steps = 0;
        if (!nown && !it && wid == 0) asm volatile("buffer_inv sc0 sc1" ::: "memory");
        if (!nown) { upto = wait_steps(ws0, lds, 0, call, wid, lane); steps = ctl[192 + lane]; REMOTE(cA, cB, 0); STB(0, 0, true, cB); STA(0, 0, true, cA); STB(0, 1, true, cB); STA(0, 1, true, cA); }

        f32x4 acc[2][2][4][2];
#pragma unroll
        for (int a = 0; a < 2; ++a)
#pragma unroll
            for (int b = 0; b < 2; ++b)
#pragma unroll
                for (int m = 0; m < 4; ++m)
#pragma unroll
                    for (int n = 0; n < 2; ++n) acc[a][b][m][n] = (f32x4){0.f, 0.f, 0.f, 0.f};
        bf16x8 At[4][2], B0[2][2], B1[2][2];
        if (wr == 1) BAR;
        WAIT_V(4); BAR;
        STB(1, 0, cs, cB); STA(1, 0, cs, cA); STB(1, 1, cs, cB);
        WAIT_V(6); BAR; CFENCE;
        steps = ctl[192 + lane];
        for (int s = 0; s < nsteps; ++s) {
            unsigned nA = cA, nB = cB; bool ns = cs;
            if (s + 1 < nsteps) {
                if (s + 1 < nown) { nA = cA + 256; nB = cB + 128 * WP; if (s == 1 && !it && wid == 0) asm volatile("buffer_inv sc0 sc1" ::: "memory"); }
                else {
                    if (s + 1 >= upto) upto = wait_steps(ws0, lds, s + 1, call, wid, lane);
                    REMOTE(nA, nB, s + 1); ns = true;
                }
            }
            LDB(B0, 0, 0); SCHED; LDA(At, 0, 0); STA(1, 1, cs, cA);
            WAIT_L(8); BAR; WAIT_L(0); SCHED; MMA(0, 0, At, B0); BAR; SCHED;
            LDB(B1, 0, 1); STB(0, 0, ns, nB);
            BAR; WAIT_L(0); SCHED; MMA(0, 1, At, B1); BAR;
            LDA(At, 0, 1); STA(0, 0, ns, nA);
            BAR; WAIT_L(0); SCHED; MMA(1, 0, At, B0); BAR; SCHED;
            STB(0, 1, ns, nB);
            WAIT_V(6); BAR; MMA(1, 1, At, B1); BAR;
            LDB(B0, 1, 0); SCHED; LDA(At, 1, 0); STA(0, 1, ns, nA);
            WAIT_L(8); BAR; WAIT_L(0); SCHED; MMA(0, 0, At, B0); BAR; SCHED;
            LDB(B1, 1, 1); STB(1, 0, ns, nB);
            BAR; WAIT_L(0); SCHED; MMA(0, 1, At, B1); BAR;
            LDA(At, 1, 1); STA(1, 0, ns, nA);
            BAR; WAIT_L(0); SCHED; MMA(1, 0, At, B0); BAR; SCHED;
            STB(1, 1, ns, nB);
            WAIT_V(6); BAR; MMA(1, 1, At, B1); BAR;
            cA = nA; cB = nB; cs = ns;
        }
        const unsigned tmo_v = __hip_atomic_load((gu32*)(ws0 + TMO_OFF), __ATOMIC_RELAXED, __HIP_MEMORY_SCOPE_AGENT);
        DRAIN_V;
        if (wr == 0) BAR;
        BAR;
        const unsigned yin = YIN_OFF, yout = XST_OFF + (call & 1) * PAR_BYTES + YOUT_REL;
        int tz = threadIdx.x; asm volatile("" : "+v"(tz));
        const unsigned fragoff = (unsigned)tz * 16;
        if (!fin) {
            const unsigned at = (aux >> 8) & 255, tile = mode == 2 ? at : rp * 16 + cp;
            const auto rt = RSRC(ws0 + ((dq == 0 ? yin : yout) + tile * TILE_B), TILE_B);
            bool bad = (unsigned)__builtin_amdgcn_readfirstlane(tmo_v) == call;
            if (mode & 1) {
                if (wid == 0) {
                    const unsigned nd = mode == 1 ? call : call * CALL_STRIDE + (p->upos[at][3] + 1) * BPU;
                    ctl[64 + lane] = nd; ctl[128 + lane] = nd; ctl[192 + lane] = mode == 1 ? (256 + at) << 8 : 1;
                    WAIT_L(0);
                }
                bad = wait_quarters(ws0, lds, 1, 0, call, wid, lane) & 16;
                const auto ri = RSRC(ws0 + ((mode == 1 ? yout : yin) + at * TILE_B), TILE_B);
                u32x4 pi[16];
#pragma unroll
                for (int i = 0; i < 16; ++i) pi[i] = __builtin_amdgcn_raw_buffer_load_b128(ri, bad ? OOB : fragoff + i * 8192, 0, 17);
#pragma unroll
                for (int i = 0; i < 16; ++i) {
                    const f32x4 v0 = acc[i >> 3][i & 1][(i >> 1) & 3][0], v1 = acc[i >> 3][i & 1][(i >> 1) & 3][1];
                    u32x4 o = {pack_bf16(v0.xy + unpack_bf16(pi[i].x)), pack_bf16(v0.zw + unpack_bf16(pi[i].y)), pack_bf16(v1.xy + unpack_bf16(pi[i].z)), pack_bf16(v1.zw + unpack_bf16(pi[i].w))};
                    if (bad) o = NAN4;
                    __builtin_amdgcn_raw_buffer_store_b128(o, rt, fragoff + i * 8192, 0, 17);
                }
            } else {
#pragma unroll
                for (int i = 0; i < 16; ++i) {
                    const f32x4 v0 = acc[i >> 3][i & 1][(i >> 1) & 3][0], v1 = acc[i >> 3][i & 1][(i >> 1) & 3][1];
                    u32x4 o = {pack_bf16(v0.xy), pack_bf16(v0.zw), pack_bf16(v1.xy), pack_bf16(v1.zw)};
                    if (bad) o = NAN4;
                    __builtin_amdgcn_raw_buffer_store_b128(o, rt, fragoff + i * 8192, 0, 17);
                }
            }
            WAIT_V(0);
            __syncthreads();
            if (threadIdx.x == 0) __hip_atomic_store((gu32*)(ws0 + TFLAG_OFF) + (mode == 2 ? 256 : 0) + tile, call, __ATOMIC_RELAXED, __HIP_MEMORY_SCOPE_AGENT);
        } else {
            if (wid == 0) {
                const unsigned in = (tz >> 2) & 1, t = (in ? (item >> 20) & 15 : rp) * 16 + (cp & 15), dm = dq ^ in;
                const unsigned up = p->upos[t][tz & 3], end = p->nent[dm] * BPC;
                const bool absent = (aux >> (2 + in)) & 1;
                ctl[64 + tz] = absent ? 0 : !dm ? call : call * CALL_STRIDE + (up >= 0xfffe ? end - PULL_LEAD : (up + 1) * BPU);
                ctl[128 + tz] = absent ? 0 : !dm ? call : call * CALL_STRIDE + end;
                ctl[192 + tz] = dm | (t << 8);
                WAIT_L(0);
            }
            const unsigned sz0 = (aux & 4) ? 0 : TILE_B, sz1 = (aux & 8) ? 0 : TILE_B;
            const auto r0 = RSRC(ws0 + (yin + (rp * 16 + (cp & 15)) * TILE_B), sz0), r1 = RSRC(ws0 + (yin + (((item >> 20) & 15) * 16 + (cp & 15)) * TILE_B), sz1);
            const int lz = tz & 63, wz = tz >> 6, fqz = lz >> 4;
            const unsigned ooff = (unsigned)(op * 256 + (wz >> 2) * 64 + (lz & 15)) * WP + colb + (wz & 3) * 64 + (fqz & 1) * 32 + (fqz >> 1) * 16;
            const auto ro = RSRC(p->out, OUT_BYTES);
            u32x4 pa[2][4] = {}, pb[2][4] = {};
#define QLOAD(q) do { const unsigned _lo = (known & 16) ? OOB : fragoff + (q) * 32768;        \
                _Pragma("unroll") for (int i = 0; i < 4; ++i) { pa[(q) & 1][i] = __builtin_amdgcn_raw_buffer_load_b128(r0, _lo + i * 8192, 0, 17); \
                                                                pb[(q) & 1][i] = __builtin_amdgcn_raw_buffer_load_b128(r1, _lo + i * 8192, 0, 17); } } while (0)
#define FINISH(q, i, a, b) do { constexpr int ai = (q) >> 1, m = ((q) & 1) * 2 + ((i) >> 1), bj = (i) & 1; \
                const f32x4 v0 = acc[ai][bj][m][0], v1 = acc[ai][bj][m][1]; \
                const unsigned o0 = pack_bf16(silu2(v0.xy + unpack_bf16(a.x) + unpack_bf16(b.x))), o1 = pack_bf16(silu2(v0.zw + unpack_bf16(a.y) + unpack_bf16(b.y))); \
                const unsigned o2 = pack_bf16(silu2(v1.xy + unpack_bf16(a.z) + unpack_bf16(b.z))), o3 = pack_bf16(silu2(v1.zw + unpack_bf16(a.w) + unpack_bf16(b.w))); \
                const auto s0 = __builtin_amdgcn_permlane16_swap(o0, o2, false, false); \
                const auto s1 = __builtin_amdgcn_permlane16_swap(o1, o3, false, false); \
                u32x4 o = {s0[0], s1[0], s0[1], s1[1]}; \
                if (known & 16) o = NAN4; \
                __builtin_amdgcn_raw_buffer_store_b128(o, ro, ooff + (unsigned)(ai * 128 + m * 16) * WP + bj * 256, 0, 17); } while (0)
#define QFIN(q) do { CFENCE; FINISH(q, 0, pa[(q) & 1][0], pb[(q) & 1][0]); FINISH(q, 1, pa[(q) & 1][1], pb[(q) & 1][1]); \
                FINISH(q, 2, pa[(q) & 1][2], pb[(q) & 1][2]); FINISH(q, 3, pa[(q) & 1][3], pb[(q) & 1][3]); CFENCE; } while (0)
            const unsigned t0 = rp * 16 + (cp & 15), t1 = ((item >> 20) & 15) * 16 + (cp & 15);
            const unsigned u0 = __builtin_amdgcn_readfirstlane(p->upos[t0][3]), u1 = __builtin_amdgcn_readfirstlane(p->upos[t1][3]);
            const auto q0 = RSRC(p->ws[u0 >= 0xfffe ? dq : 0] + ((u0 >= 0xfffe ? yout : yin) + t0 * TILE_B), sz0);
            const auto q1 = RSRC(p->ws[u1 >= 0xfffe ? dq ^ 1 : 0] + ((u1 >= 0xfffe ? yout : yin) + t1 * TILE_B), sz1);
            const bool second = ((aux & 8) ? u0 : u1) == 0xffff;
            int known = wait_quarters(ws0, lds, 1, 0, call, wid, lane);
            QLOAD(0);
            const bool ahead = known & 2;
            if (ahead) QLOAD(1);
            QFIN(0);
            if (!ahead) { known = wait_quarters(ws0, lds, 2, 1, call, wid, lane); QLOAD(1); }
            QFIN(1);
            for (int todo = 12; todo;) {
                const bool fresh = !(known & todo);
                if (fresh) known = wait_quarters(ws0, lds, todo, todo == 12 ? 2 : 3, call, wid, lane);
                if ((known & todo & 8) && (!second || !(known & todo & 4))) {
                    if (second && fresh && (known & 32)) {
                        const unsigned long long tb = __builtin_amdgcn_s_memrealtime();
                        while (__builtin_amdgcn_s_memrealtime() - tb < PULL_DELAY) __builtin_amdgcn_s_sleep(2);
                    }
                    const unsigned lo = (known & 16) ? OOB : fragoff + 3 * 32768;
#define PLOAD(i) do { CFENCE; pa[0][i] = __builtin_amdgcn_raw_buffer_load_b128(q0, lo + (i) * 8192, 0, 17); pb[0][i] = __builtin_amdgcn_raw_buffer_load_b128(q1, lo + (i) * 8192, 0, 17); CFENCE; } while (0)
#define PWAIT(i) asm volatile("" : "+v"(pa[0][i]), "+v"(pb[0][i]))
                    PLOAD(0);
                    PWAIT(0); PLOAD(1); FINISH(3, 0, pa[0][0], pb[0][0]);
                    PWAIT(1); PLOAD(2); FINISH(3, 1, pa[0][1], pb[0][1]);
                    PWAIT(2); PLOAD(3); FINISH(3, 2, pa[0][2], pb[0][2]);
                    FINISH(3, 3, pa[0][3], pb[0][3]);
#undef PLOAD
#undef PWAIT
                    todo &= ~8;
                } else { QLOAD(2); QFIN(2); todo &= ~4; }
            }
#undef QFIN
#undef FINISH
#undef QLOAD
        }
    }
}

__global__ void __launch_bounds__(512, 2) fused_gemm_rs(const Params args) {
    extern __shared__ __attribute__((aligned(16))) unsigned char shm[];
    KP p = (KP)__builtin_amdgcn_kernarg_segment_ptr();
    const int wid = __builtin_amdgcn_readfirstlane(threadIdx.x >> 6);
    const int d = 1 + (wid + blockIdx.x) % (NR - 1);
    unsigned long long w0 = p->ws[0];
    Pre pre; pre.x = p->x; pre.w = p->w; pre.wsd = p->ws[d]; pre.slot0 = p->lbase[d]; pre.nent = p->nent[d]; pre.nhead = p->nhead[d];
#pragma unroll
    for (int j = 0; j < NPUSH; ++j) pre.he[j] = p->head[d][blockIdx.x & 31][j];
    unsigned first = p->items[blockIdx.x][0], qi = p->quick[blockIdx.x][0], qe = p->quick[blockIdx.x][1], rank = p->rank;
    asm volatile("" : "+s"(w0), "+s"(pre.x), "+s"(pre.w), "+s"(pre.wsd), "+s"(pre.slot0), "+s"(pre.nent), "+s"(pre.nhead), "+s"(pre.he[0]),
                      "+s"(first), "+s"(qi), "+s"(qe), "+s"(rank));
    gchar* ws0 = (gchar*)w0;
    gu32* ctr = (gu32*)(ws0 + CTR_OFF + blockIdx.x * 64) + wid;
    const unsigned prev = __hip_atomic_load(ctr, __ATOMIC_RELAXED, __HIP_MEMORY_SCOPE_AGENT);
    if (blockIdx.x < (unsigned)NC) copy_role(p, ws0, ctr, prev, d, pre);
    else {
        const unsigned long long tb = __builtin_amdgcn_s_memrealtime();
        const unsigned call = (first & 0x10000000u) ? export_role(p, ws0, ctr, prev, (first >> 18) & 63, qe, rank, pre.x, pre.w)
                            : qi ? export_quick(ws0, ctr, prev, qi - 1, qe, pre.x, pre.w) : next_call(ctr, prev);
        if (first & 0x20000000u)
            while (__builtin_amdgcn_s_memrealtime() - tb < (unsigned long long)LAZY_TICKS * (1 + ((first >> 24) & 3))) __builtin_amdgcn_s_sleep(32);
        gemm_role(p, ws0, (LAS unsigned char*)shm, call);
    }
}

void build_tables(Params& p) {
    memset(p.ent, 0, sizeof p.ent); memset(p.gate, 0xff, sizeof p.gate);
    memset(p.posx, 0, sizeof p.posx); memset(p.posw, 0, sizeof p.posw); memset(p.upos, 0xff, sizeof p.upos);
    for (int d = 0; d < NR; ++d) {
        p.nent[d] = SCHED_NENT[d];
        for (int i = 0; i < SCHED_NENT[d]; ++i) {
            const unsigned e = p.ent[d][i] = SCHED_ENT[d][i], type = e >> 30, round = (e >> 12) & 3, panel = (e >> 8) & 15, df = (e >> 5) & 7;
            if (type == 0) p.posx[d][panel][round] = (unsigned char)i;
            else if (type == 1) p.posw[d][panel][round] = (unsigned short)((d << 8) | i);
            else if (type == 2) p.posw[df ^ d][panel][round] = (unsigned short)((d << 8) | i);
        }
    }
    for (int d = 0, nb = 0; d < NR; ++d) {
        p.lbase[d] = nb;
        for (int i = 0; i < SCHED_NENT[d]; ++i) nb += (p.ent[d][i] >> 30) < 3;
    }
    for (int d = 1; d < NR; ++d) {
        int ng = 0;
        for (int i = 0; i < SCHED_NENT[d]; ++i) {
            const unsigned e = p.ent[d][i], type = e >> 30, round = (e >> 12) & 3, panel = (e >> 8) & 15, df = (e >> 5) & 7;
            if (type == 2) {
                p.gate[d][ng++] = (unsigned)(2 * i) | (df << 9) | (((p.posw[df][panel][round] & 255u) + 1) << 16);
                p.ent[d][i] |= (p.lbase[df] + (p.posw[df][panel][round] & 255u)) << 16;
            }
            else if (type == 3)
                for (int h = 0; h < 2; ++h) {
                    const unsigned u = (e >> (h ? 0 : 10)) & 1023;
                    p.upos[u >> 2][u & 3] = (unsigned short)(2 * i + h);
                    if ((u & 3) == 0) p.gate[d][ng++] = (unsigned)(2 * i + h) | ((u >> 2) << 16);
                }
        }
    }
    memset(p.head, 0xff, sizeof p.head); memset(p.nhead, 0, sizeof p.nhead);
    for (int d = 2; d < NR; ++d)
        for (int j = 0; j < NPUSH; ++j)
            for (int b = 0; b < NS; ++b) {
                if (j == 0 && (d - 1 + 7 * 8 - b) % 7 >= 4) continue;
                const unsigned g = p.nhead[d]++;
                const unsigned e = p.ent[d][g / BPC], rcv = p.rank ^ d, isw = e >> 30, round = (e >> 12) & 3, panel = (e >> 8) & 15, df = (e >> 5) & 7, op = (e >> 3) & 3;
                const unsigned ksub = 2 * round + (((round & 1) ? d ^ (d >> 1) : d) & 1);
                const unsigned corner = isw ? ksub * 128 * WP + (rcv >> 2) * 8192 + panel * 512 : ((rcv ^ df) * 1024 + op * 256) * XP + ksub * 256;
                p.head[d][b][j] = (corner + image_src(isw, (g % BPC) * CU_, 0)) | isw | (g << 24);
            }
    memset(p.epos, 0, sizeof p.epos); memset(p.exl, 0, sizeof p.exl);
    int where[512]; for (int i = 0; i < 512; ++i) where[i] = 0;
    p.nexp = 0; p.pad2 = 0;
    for (int k = 0; k < MAXE; ++k)
        for (int d = NR - 1; d >= 1; --d) {
            const int i = d == 1 ? k - 14 : k;
            if (i < (int)p.nhead[d] / BPC || i >= (int)p.nent[d] || (p.ent[d][i] >> 30) > 1) continue;
            const unsigned e = p.ent[d][i], type = e >> 30, round = (e >> 12) & 3, panel = (e >> 8) & 15, df = (e >> 5) & 7, op = (e >> 3) & 3;
            const unsigned ksub = d == 1 ? 2 * XR + round : 2 * round + (((round & 1) ? d ^ (d >> 1) : d) & 1);
            const unsigned id = (type << 8) | (ksub << 5) | (type ? (d == 1 || d >= 4) * 16 + panel : (df ^ d) * 4 + op);
            if (!where[id]) { p.exl[p.nexp] = (unsigned short)id; where[id] = ++p.nexp; }
            p.epos[d][i] = (unsigned short)where[id];
            p.ent[d][i] |= (unsigned)(where[id] - 1) << 16;
        }
    for (int t = 0; t < 256; ++t) if (p.upos[t][3] == 0xffff && p.upos[t][2] != p.upos[t][1] + 1) p.upos[t][3] = 0xfffe;
    for (int t = 0; t < 256; ++t) if (p.upos[t][0] == 0xffff) for (int q = 0; q < 4; ++q) p.upos[t][q] = 0;
    for (int b = 0; b < GRID; ++b) { p.items[b][0] = SCHED_ITEMS[b][0]; p.items[b][1] = SCHED_ITEMS[b][1]; p.aux[b][0] = SCHED_AUX[b][0]; p.aux[b][1] = SCHED_AUX[b][1]; }
    for (int cls = 2, ne = 0; cls >= 0; --cls)
        for (int b = GRID - 1; b >= NC; --b)
            if ((p.items[b][0] & 0x20000000u) && (int)((p.items[b][0] >> 24) & 3) == cls && ne < NE) { p.items[b][0] |= 0x10000000u | (unsigned)(ne++ << 18); }
    memset(p.quick, 0, sizeof p.quick);
    for (int b = NC; b < GRID; ++b) if (p.items[b][0] & 0x10000000u) p.quick[b][1] = p.exl[NQ + ((p.items[b][0] >> 18) & 63)] | (p.nexp << 16);
    for (int pass = 0, nq = 0; pass < 2; ++pass)
        for (int b = NC; b < GRID && nq < 4 * NQ; ++b)
            if (!(p.items[b][0] & 0x10000000u) && !p.quick[b][0] && (pass || (p.items[b][0] & 0x20000000u))) {
                const unsigned e = p.exl[nq >> 2], w = e >> 8, ks = (e >> 5) & 7, pn = e & 31;
                const unsigned c = w ? ks * 128 * WP + (((p.rank >> 2) * 16) ^ pn) * 512 : ((p.rank ^ (pn >> 2)) * 4 + (pn & 3)) * 256 * XP + ks * 256;
                p.quick[b][1] = (c + image_src(w, (nq & 3) * 16, 0)) | w; p.quick[b][0] = ++nq;
            }
}

}

extern "C" void kernel_launch(int rank, int nranks, ncclComm_t comm, hipStream_t stream,
                              void* const* d_in, const int* in_sizes, int n_in,
                              void* d_out, int out_size, const int* in_dtypes, int out_dtype,
                              void* const* d_out_all, void* const* d_ws_all, size_t ws_bytes) {
    (void)comm; (void)in_dtypes; (void)out_dtype; (void)d_out_all;
    if (nranks != NR || (unsigned)rank >= (unsigned)NR || n_in != 2 || in_sizes[0] != 8192 * 1024 || in_sizes[1] != 1024 * 8192 ||
        out_size != 1024 * 8192 || ws_bytes < WS_NEED) { fprintf(stderr, "unexpected problem shape\n"); return; }
    Params p;
    p.x = (unsigned long long)d_in[0]; p.w = (unsigned long long)d_in[1]; p.out = (unsigned long long)d_out;
    for (int d = 0; d < NR; ++d) p.ws[d] = (unsigned long long)d_ws_all[rank ^ d];
    p.rank = rank; p.pad = 0;
    build_tables(p);
    HIP_CHECK(hipFuncSetAttribute((const void*)fused_gemm_rs, hipFuncAttributeMaxDynamicSharedMemorySize, LDS_BYTES));
    hipLaunchKernelGGL(fused_gemm_rs, dim3(GRID), dim3(512), LDS_BYTES, stream, p);
    HIP_CHECK(hipGetLastError());
}
```
